```python
import numpy as np

kernel_name = "gpu_gemm_rs_m4096_k4096_n16384_bf16_none"


ABSMAX_RTOL = 0.008

COMPUTE_DTYPE = "bfloat16"
EPILOGUE = "none"
IN_QUANT = None
OUT_QUANT = None
IN_DTYPES = {"x": "bfloat16", "w_mat": "bfloat16"}
WORLD_SIZE = 8

def _to_bf16(x):
    u = np.ascontiguousarray(x, dtype=np.float32).view(np.uint32)
    return ((u + 0x7FFF + ((u >> 16) & 1)) & 0xFFFF0000).view(np.float32)

def setup_inputs():

    x = _to_bf16(np.random.randn(4096, 4096).astype(np.float32))
    w_mat = _to_bf16((np.random.randn(4096, 16384) / np.sqrt(4096)).astype(np.float32))
    return {"x": x, "w_mat": w_mat}

def _epilogue(y):
    return y

def reference(x, w_mat):
    y = (x @ w_mat).astype(np.float32)
    return _epilogue(y)
```

```cpp
#include <hip/hip_runtime.h>
#include <rccl/rccl.h>
#include <cstdio>
#include <cstdint>

#define HIP_CHECK(expr) do { hipError_t _e = (expr); if (_e != hipSuccess) { \
    fprintf(stderr, "HIP error %s at %s:%d\n", hipGetErrorString(_e), __FILE__, __LINE__); \
    return; } } while (0)

namespace {

#define LAS __attribute__((address_space(3)))
typedef short bf16x8 __attribute__((ext_vector_type(8)));
typedef short s16x4 __attribute__((ext_vector_type(4)));
typedef float f32x4 __attribute__((ext_vector_type(4)));
typedef float f32x2 __attribute__((ext_vector_type(2)));
typedef unsigned u32x4 __attribute__((ext_vector_type(4)));
typedef __bf16 bf16x2_t __attribute__((ext_vector_type(2)));

constexpr int NR = 8;
constexpr unsigned XP = 1024;
constexpr unsigned WP = 32768;
constexpr unsigned X_BYTES = 4096u * XP;
constexpr unsigned W_BYTES = 512u * WP;
constexpr unsigned O_BYTES = 512u * WP;
constexpr unsigned UNIT = 32768;
constexpr unsigned HTB = 16384;

constexpr unsigned CTR_OFF = 0;
constexpr unsigned TMO_OFF = 16384;
constexpr unsigned XF_OFF = 32768;
constexpr unsigned WF_OFF = 36864;
constexpr unsigned PF_OFF = 49152;
constexpr unsigned STG_OFF = 1u << 20;
constexpr unsigned XS_REL = 0, WS_REL = 32u << 20, PS_REL = 48u << 20, STG_BYTES = 64u << 20;
constexpr unsigned WS_NEED = STG_OFF + 2 * STG_BYTES;

constexpr int NCOPY = 112, NGEMM = 128, NBLK = NCOPY + NGEMM;
constexpr int LAG = 3;
constexpr int NUNITS = 176;
constexpr unsigned LDS_BYTES = 131072 + 64;
constexpr unsigned long long DEADLINE_TICKS = 40000000ull;

struct Args {
    const char* x; const char* w; char* out;
    char* ws[NR];
    int rank;
};

__device__ __forceinline__ int lds_byte(int r, int c) {
    const int st = (r >> 4) * 2 + (c >> 5), rr = r & 15, cc = c & 31, ob = rr * 64 + cc * 2;
    return st * 1024 + (ob ^ (((ob >> 9) & 1) << 5));
}
__device__ __forceinline__ void stage_rc(int b, int& R, int& C) {
    const int st = b / 1024, sb = b % 1024, swz = sb ^ (((sb >> 9) & 1) << 5);
    R = (st >> 1) * 16 + swz / 64; C = (st & 1) * 32 + (swz % 64) / 2;
}
__device__ __forceinline__ unsigned pack_bf16(float lo, float hi) {
    f32x2 v = {lo, hi};
    return __builtin_bit_cast(unsigned, __builtin_convertvector(v, bf16x2_t));
}
__device__ __forceinline__ __amdgpu_buffer_rsrc_t mk_rsrc(const void* p, unsigned bytes) {
    return __builtin_amdgcn_make_buffer_rsrc((void*)p, 0, (int)bytes, 0x00020000);
}
__device__ __forceinline__ unsigned ld_sys(const unsigned* p) { return __hip_atomic_load(p, __ATOMIC_RELAXED, __HIP_MEMORY_SCOPE_SYSTEM); }
__device__ __forceinline__ unsigned ld_agent(const unsigned* p) { return __hip_atomic_load(p, __ATOMIC_RELAXED, __HIP_MEMORY_SCOPE_AGENT); }
__device__ __forceinline__ void st_sys(unsigned* p, unsigned v) { __hip_atomic_store(p, v, __ATOMIC_RELAXED, __HIP_MEMORY_SCOPE_SYSTEM); }
__device__ __forceinline__ void st_agent(unsigned* p, unsigned v) { __hip_atomic_store(p, v, __ATOMIC_RELAXED, __HIP_MEMORY_SCOPE_AGENT); }

#define WAIT_V(n) asm volatile("s_waitcnt vmcnt(" #n ")" ::: "memory")
#define WAIT_L(n) asm volatile("s_waitcnt lgkmcnt(" #n ")" ::: "memory")
#define BAR __builtin_amdgcn_s_barrier()
#define SCHED __builtin_amdgcn_sched_barrier(0)
#define CFENCE asm volatile("" ::: "memory")

__device__ __forceinline__ void decode_unit(int n, int& kind, int& step, int& idx) {
    constexpr int P1 = 15 * LAG, P2 = P1 + 22 * (8 - LAG);
    int s, r;
    if (n < P1) { s = n / 15; r = n - s * 15; }
    else if (n < P2) { const int m = n - P1; const int q = m / 22; s = LAG + q; r = m - q * 22; }
    else { const int m = n - P2; const int q = m / 7; s = 8 + q; r = 15 + (m - q * 7); }
    if (r < 8) { kind = 0; step = s; idx = r; }
    else if (r < 15) { kind = 1; step = s; idx = r - 8; }
    else { kind = 2; step = s - LAG; idx = r - 15; }
}

__global__ __launch_bounds__(512, 2) void fused_gemm_rs(Args a) {
    extern __shared__ __attribute__((aligned(16))) unsigned char shm[];
    LAS unsigned char* lds = (LAS unsigned char*)shm;
    const int tid = threadIdx.x, wid = __builtin_amdgcn_readfirstlane(tid >> 6), lane = tid & 63;
    const int b = blockIdx.x, rank = a.rank & 7;
    char* const myws = a.ws[rank];
    unsigned* const tmo = (unsigned*)(myws + TMO_OFF);

    unsigned* const ctr = (unsigned*)(myws + CTR_OFF + (unsigned)b * 64u);
    const unsigned call = (unsigned)__builtin_amdgcn_readfirstlane((int)ld_agent(ctr)) + 1u;
    WAIT_V(0);
    __syncthreads();
    if (tid == 0) st_agent(ctr, call);
    const unsigned stg = STG_OFF + (call & 1u) * STG_BYTES;

    unsigned voffA, voffB;
    { int R, C; stage_rc(tid * 16, R, C); voffA = (unsigned)R * XP + (unsigned)C * 2u;
      const int row = tid >> 4, chp = tid & 15, f = ((row & 3) << 2) | ((row >> 2) & 3);
      voffB = (unsigned)row * WP + (unsigned)(chp ^ f) * 16u; }
    const unsigned voffS = (unsigned)tid * 16u;

    const bool is_copy = b < NCOPY;
    int mi, nj, colrank;
    if (is_copy) {
        const int dd = 1 + b % 7, c = b / 7;
        const int dest = (rank + dd) & 7;
        colrank = dest; mi = rank * 2 + (c >> 3); nj = c & 7;
        const __amdgpu_buffer_rsrc_t rD = mk_rsrc(a.ws[dest], WS_NEED);
        char* const dws = a.ws[dest];
        u32x4 r[4], rn[4];
        unsigned doff, foff;
        auto load_unit = [&](int n, u32x4 (&v)[4], unsigned& dsto, unsigned& flo) {
            int kind, step, idx; decode_unit(n, kind, step, idx);
            if (kind == 0) {
                const __amdgpu_buffer_rsrc_t rs = mk_rsrc(a.w, W_BYTES);
                const unsigned so = (unsigned)step * 64u * WP + (unsigned)(dest * 2048 + idx * 256) * 2u;
#pragma unroll
                for (int h = 0; h < 2; ++h)
#pragma unroll
                    for (int i = 0; i < 2; ++i)
                        v[h * 2 + i] = __builtin_amdgcn_raw_buffer_load_b128(rs, voffB, so + h * 256u + i * 32u * WP, 0);
                dsto = stg + WS_REL + (unsigned)((rank * 8 + step) * 8 + idx) * UNIT;
                flo = WF_OFF + (unsigned)(idx * 64 + rank * 8 + step) * 4u;
            } else {
                const int tm = ((rank + 1 + idx) & 7) * 2 + (kind - 1);
                const __amdgpu_buffer_rsrc_t rs = mk_rsrc(a.x, X_BYTES);
                const unsigned so = (unsigned)tm * 256u * XP + (unsigned)step * 128u;
#pragma unroll
                for (int h = 0; h < 2; ++h)
#pragma unroll
                    for (int i = 0; i < 2; ++i)
                        v[h * 2 + i] = __builtin_amdgcn_raw_buffer_load_b128(rs, voffA, so + h * 128u * XP + i * 64u * XP, 0);
                dsto = stg + XS_REL + (unsigned)((rank * 8 + step) * 16 + tm) * UNIT;
                flo = XF_OFF + (unsigned)(tm * 64 + rank * 8 + step) * 4u;
            }
        };
        load_unit(c, r, doff, foff);
#pragma unroll 1
        for (int k = 0; k < NUNITS / 16; ++k) {
#pragma unroll
            for (int u = 0; u < 4; ++u)
                __builtin_amdgcn_raw_buffer_store_b128(r[u], rD, voffS, doff + (unsigned)u * 8192u, 17);
            unsigned doffn = doff, foffn = foff;
            if (k + 1 < NUNITS / 16) load_unit(c + 16 * (k + 1), rn, doffn, foffn);
            WAIT_V(0);
            asm volatile("" : "+v"(rn[0]), "+v"(rn[1]), "+v"(rn[2]), "+v"(rn[3]));
            __syncthreads();
            if (tid == 0) st_sys((unsigned*)(dws + foff), call);
#pragma unroll
            for (int u = 0; u < 4; ++u) r[u] = rn[u];
            doff = doffn; foff = foffn;
        }
    } else {
        const int g = b - NCOPY, xcd = g & 7, idx = g >> 3;
        mi = 4 * (xcd >> 1) + (idx & 3); nj = 4 * (xcd & 1) + (idx >> 2); colrank = rank;
    }
    const int rb = mi >> 1, hh = mi & 1;

    const int wr = wid >> 2, wc = wid & 3, fr = lane & 15, fq = lane >> 4;
    const unsigned ldsw = (unsigned)wid * 1024u;
    const int aoff = lds_byte(wr * 64 + fr, fq * 8);
    unsigned boff[2][2];
    { const int q = fr >> 2, p = lane & 3;
#pragma unroll
      for (int n = 0; n < 2; ++n)
#pragma unroll
          for (int t = 0; t < 2; ++t) {
              const int ch = 4 * wc + 2 * n + (p >> 1), f = (q << 2) | ((2 * fq + t) & 3);
              boff[n][t] = (unsigned)(size_t)lds + 65536u + 256u * (unsigned)(8 * fq + 4 * t + q) + 16u * (unsigned)(ch ^ f) + 8u * (unsigned)(p & 1);
          } }
#define SA(bb, h) (((bb) * 2 + (h)) * HTB)
#define SB(bb, h) ((4 + (bb) * 2 + (h)) * HTB)
#define SBR(bb, h) (((bb) * 2 + (h)) * HTB)
#define STAGE(bufoff, rs, voff, soff, istr) do { _Pragma("unroll") for (int _i = 0; _i < 2; ++_i) \
        __builtin_amdgcn_raw_ptr_buffer_load_lds(rs, (LAS void*)(lds + (bufoff) + ldsw + _i * 8192), 16, voff, (soff) + _i * (istr), 0, 16); } while (0)
#define LDA(dst, bb, h) do { _Pragma("unroll") for (int m = 0; m < 4; ++m) _Pragma("unroll") for (int k = 0; k < 2; ++k) \
        dst[m][k] = *(const LAS bf16x8*)(lds + SA(bb, h) + aoff + m * 2048 + k * 1024); } while (0)
#define TRR(d, addr, off) asm volatile("ds_read_b64_tr_b16 %0, %1 offset:%2" : "=&v"(d) : "v"(addr), "i"(off) : "memory")
#define LDB(dst, bb, h) do { _Pragma("unroll") for (int n = 0; n < 2; ++n) _Pragma("unroll") for (int k = 0; k < 2; ++k) { \
        TRR(dst[n][k][0], boff[n][0], SBR(bb, h) + k * 8192); TRR(dst[n][k][1], boff[n][1], SBR(bb, h) + k * 8192); } } while (0)
#define PKB(v) __builtin_shufflevector((v)[0], (v)[1], 0, 1, 2, 3, 4, 5, 6, 7)
#define MMA(ai, bj, At, Bt) do { SCHED; __builtin_amdgcn_s_setprio(1); _Pragma("unroll") for (int m = 0; m < 4; ++m) _Pragma("unroll") for (int n = 0; n < 2; ++n) \
        _Pragma("unroll") for (int k = 0; k < 2; ++k) acc[ai][bj][m][n] = __builtin_amdgcn_mfma_f32_16x16x32_bf16(PKB(Bt[n][k]), At[m][k], acc[ai][bj][m][n], 0, 0, 0); \
        __builtin_amdgcn_s_setprio(0); } while (0)

    f32x4 acc[2][2][4][2];
#pragma unroll
    for (int i0 = 0; i0 < 2; ++i0)
#pragma unroll
        for (int i1 = 0; i1 < 2; ++i1)
#pragma unroll
            for (int m = 0; m < 4; ++m)
#pragma unroll
                for (int n = 0; n < 2; ++n) acc[i0][i1][m][n] = (f32x4){0.f, 0.f, 0.f, 0.f};
    bf16x8 At[4][2]; s16x4 B0[2][2][2], B1[2][2][2];

    const int np = (rb == rank) ? 7 : 6, npmul = (rb == rank) ? 37 : 43;
    const int eskip = (rb == rank) ? 8 : ((rb - rank - 1) & 7);
    const int nph = is_copy ? 1 : 2;
#pragma unroll 1
    for (int ph = 0; ph < nph; ++ph) {
        const bool own = (ph == 0);
        const int nt = own ? 8 : np * 8;
        const __amdgpu_buffer_rsrc_t rA = mk_rsrc(own ? a.x : (const char*)myws, own ? X_BYTES : WS_NEED);
        const __amdgpu_buffer_rsrc_t rB = mk_rsrc(own ? a.w : (const char*)myws, own ? W_BYTES : WS_NEED);
        const unsigned vA = own ? voffA : voffS, vB = own ? voffB : voffS;
        const unsigned iA = own ? 64u * XP : 8192u, iB = own ? 32u * WP : 8192u;
        const unsigned hA = own ? 128u * XP : HTB, hB = own ? 256u : HTB;
        auto step_src = [&](int t) -> int {
            const int s = (t * npmul) >> 8, q = t - s * np, e = q + (q >= eskip ? 1 : 0);
            return ((rb - 1 - e) & 7) * 8 + s;
        };
        auto addrA = [&](int t) -> unsigned {
            return own ? (unsigned)mi * 256u * XP + (unsigned)t * 128u
                       : stg + XS_REL + (unsigned)(step_src(t) * 16 + mi) * UNIT;
        };
        auto addrB = [&](int t) -> unsigned {
            return own ? (unsigned)t * 64u * WP + (unsigned)(colrank * 2048 + nj * 256) * 2u
                       : stg + WS_REL + (unsigned)(step_src(t) * 8 + nj) * UNIT;
        };
        int avail = own ? 64 : 0;
        const int myflag = step_src(lane < nt ? lane : 0);
        const unsigned* const xf = (const unsigned*)(myws + XF_OFF) + mi * 64 + myflag;
        const unsigned* const wf = (const unsigned*)(myws + WF_OFF) + nj * 64 + myflag;
        LAS volatile int* const lavail = (LAS volatile int*)(lds + 131072);
        auto ensure = [&](int need) {
            if (avail < need) {
                if (wid == 0) {
                    const unsigned long long t0 = __builtin_amdgcn_s_memrealtime();
                    int pre;
                    for (;;) {
                        const unsigned fx = ld_sys(xf), fw = ld_sys(wf);
                        const bool ok = (lane >= nt) || ((int)(fx - call) >= 0 && (int)(fw - call) >= 0);
                        const unsigned long long mask = __builtin_amdgcn_ballot_w64(ok);
                        pre = (~mask == 0ull) ? 64 : __builtin_ctzll(~mask);
                        if (pre >= need) break;
                        if ((unsigned)__builtin_amdgcn_readfirstlane((int)ld_agent(tmo)) == call ||
                            __builtin_amdgcn_s_memrealtime() - t0 > DEADLINE_TICKS) {
                            if (lane == 0) st_agent(tmo, call);
                            pre = 64; break;
                        }
                        __builtin_amdgcn_s_sleep(2);
                    }
                    if (lane == 0) *lavail = pre;
                    WAIT_V(0); WAIT_L(0);
                }
                CFENCE; BAR; CFENCE;
                avail = __builtin_amdgcn_readfirstlane(*lavail);
            }
        };

        ensure(2);
        { const unsigned a0 = addrA(0), b0 = addrB(0), a1 = addrA(1), b1 = addrB(1);
          STAGE(SB(0, 0), rB, vB, b0, iB); STAGE(SA(0, 0), rA, vA, a0, iA); STAGE(SB(0, 1), rB, vB, b0 + hB, iB); STAGE(SA(0, 1), rA, vA, a0 + hA, iA);
          if (wr == 1) BAR;
          WAIT_V(4); BAR;
          STAGE(SB(1, 0), rB, vB, b1, iB); STAGE(SA(1, 0), rA, vA, a1, iA); STAGE(SB(1, 1), rB, vB, b1 + hB, iB);
          WAIT_V(6); BAR; }
        unsigned a1 = addrA(1);
#pragma unroll 1
        for (int t = 0; t < nt; t += 2) {
            const bool last = (t == nt - 2);
            const int t2 = last ? 0 : t + 2, t3 = last ? 1 : t + 3;
            if (!last) ensure(t + 3);
            const unsigned a2 = addrA(t2), b2 = addrB(t2);
            LDB(B0, 0, 0); SCHED; LDA(At, 0, 0); STAGE(SA(1, 1), rA, vA, a1 + hA, iA);
            WAIT_L(8); BAR; WAIT_L(0); MMA(0, 0, At, B0); BAR; SCHED;
            LDB(B1, 0, 1); STAGE(SB(0, 0), rB, vB, b2, iB);
            BAR; WAIT_L(0); MMA(0, 1, At, B1); BAR;
            LDA(At, 0, 1); STAGE(SA(0, 0), rA, vA, a2, iA);
            BAR; WAIT_L(0); MMA(1, 0, At, B0); BAR; SCHED;
            STAGE(SB(0, 1), rB, vB, b2 + hB, iB);
            WAIT_V(6); BAR; MMA(1, 1, At, B1); BAR;
            if (!last) ensure(t + 4);
            const unsigned a3 = addrA(t3), b3 = addrB(t3);
            LDB(B0, 1, 0); SCHED; LDA(At, 1, 0); STAGE(SA(0, 1), rA, vA, a2 + hA, iA);
            WAIT_L(8); BAR; WAIT_L(0); MMA(0, 0, At, B0); BAR; SCHED;
            LDB(B1, 1, 1); STAGE(SB(1, 0), rB, vB, b3, iB);
            BAR; WAIT_L(0); MMA(0, 1, At, B1); BAR;
            LDA(At, 1, 1); STAGE(SA(1, 0), rA, vA, a3, iA);
            BAR; WAIT_L(0); MMA(1, 0, At, B0); BAR; SCHED;
            STAGE(SB(1, 1), rB, vB, b3 + hB, iB);
            WAIT_V(6); BAR; MMA(1, 1, At, B1); BAR;
            a1 = a3;
        }
        WAIT_V(0);
        if (wr == 0) BAR;
        BAR;
    }

    if (!is_copy && rb != rank) {
        const bool bad = (unsigned)__builtin_amdgcn_readfirstlane((int)ld_agent(tmo)) == call;
        const __amdgpu_buffer_rsrc_t rP = mk_rsrc(a.ws[rb], WS_NEED);
        const unsigned slot = (unsigned)((rank * 2 + hh) * 8 + nj);
        const unsigned base = stg + PS_REL + slot * 131072u + (unsigned)wid * 16384u;
#pragma unroll
        for (int ai = 0; ai < 2; ++ai)
#pragma unroll
            for (int bj = 0; bj < 2; ++bj)
#pragma unroll
                for (int m = 0; m < 4; ++m) {
                    const f32x4 v0 = acc[ai][bj][m][0], v1 = acc[ai][bj][m][1];
                    u32x4 w; w.x = pack_bf16(v0[0], v0[1]); w.y = pack_bf16(v0[2], v0[3]); w.z = pack_bf16(v1[0], v1[1]); w.w = pack_bf16(v1[2], v1[3]);
                    if (bad) w = (u32x4){0x7FC07FC0u, 0x7FC07FC0u, 0x7FC07FC0u, 0x7FC07FC0u};
                    __builtin_amdgcn_raw_buffer_store_b128(w, rP, (unsigned)lane * 16u, base + (unsigned)((ai * 2 + bj) * 4 + m) * 1024u, 17);
                }
        WAIT_V(0);
        if (lane == 0) st_sys((unsigned*)(a.ws[rb] + PF_OFF) + slot * 32u + wid, call);
        return;
    }
    if (is_copy) {
        const unsigned slot = (unsigned)((colrank * 2 + hh) * 8 + nj);
        const unsigned* const pf = (const unsigned*)(myws + PF_OFF) + slot * 32u + wid;
        const unsigned long long t0 = __builtin_amdgcn_s_memrealtime();
        while ((int)((unsigned)__builtin_amdgcn_readfirstlane((int)ld_sys(pf)) - call) < 0) {
            if ((unsigned)__builtin_amdgcn_readfirstlane((int)ld_agent(tmo)) == call ||
                __builtin_amdgcn_s_memrealtime() - t0 > DEADLINE_TICKS) {
                if (lane == 0) st_agent(tmo, call);
                break;
            }
            __builtin_amdgcn_s_sleep(2);
        }
        CFENCE;
        const __amdgpu_buffer_rsrc_t rP = mk_rsrc(myws, WS_NEED);
        const unsigned base = stg + PS_REL + slot * 131072u + (unsigned)wid * 16384u;
        u32x4 pv[16];
#pragma unroll
        for (int u = 0; u < 16; ++u) pv[u] = __builtin_amdgcn_raw_buffer_load_b128(rP, (unsigned)lane * 16u, base + (unsigned)u * 1024u, 17);
#pragma unroll
        for (int ai = 0; ai < 2; ++ai)
#pragma unroll
            for (int bj = 0; bj < 2; ++bj)
#pragma unroll
                for (int m = 0; m < 4; ++m) {
                    const u32x4 w = pv[(ai * 2 + bj) * 4 + m];
                    f32x4& v0 = acc[ai][bj][m][0]; f32x4& v1 = acc[ai][bj][m][1];
                    v0[0] += __uint_as_float(w.x << 16); v0[1] += __uint_as_float(w.x & 0xFFFF0000u);
                    v0[2] += __uint_as_float(w.y << 16); v0[3] += __uint_as_float(w.y & 0xFFFF0000u);
                    v1[0] += __uint_as_float(w.z << 16); v1[1] += __uint_as_float(w.z & 0xFFFF0000u);
                    v1[2] += __uint_as_float(w.w << 16); v1[3] += __uint_as_float(w.w & 0xFFFF0000u);
                }
    }
    {
        const bool bad = (unsigned)__builtin_amdgcn_readfirstlane((int)ld_agent(tmo)) == call;
        const __amdgpu_buffer_rsrc_t rO = mk_rsrc(a.out, O_BYTES);
        const unsigned vo = (unsigned)(wr * 64 + fr) * WP + (unsigned)(wc * 32 + ((fq & 1) << 4) + ((fq >> 1) << 3)) * 2u;
        const unsigned so = (unsigned)hh * 256u * WP + (unsigned)(colrank * 2048 + nj * 256) * 2u;
#pragma unroll
        for (int ai = 0; ai < 2; ++ai)
#pragma unroll
            for (int m = 0; m < 4; ++m)
#pragma unroll
                for (int bj = 0; bj < 2; ++bj) {
                    const f32x4 v0 = acc[ai][bj][m][0], v1 = acc[ai][bj][m][1];
                    const unsigned ax = pack_bf16(v0[0], v0[1]), ay = pack_bf16(v0[2], v0[3]);
                    const unsigned bx = pack_bf16(v1[0], v1[1]), by = pack_bf16(v1[2], v1[3]);
                    const auto sx = __builtin_amdgcn_permlane16_swap(ax, bx, false, false);
                    const auto sy = __builtin_amdgcn_permlane16_swap(ay, by, false, false);
                    u32x4 w = {sx[0], sy[0], sx[1], sy[1]};
                    if (bad) w = (u32x4){0x7FC07FC0u, 0x7FC07FC0u, 0x7FC07FC0u, 0x7FC07FC0u};
                    __builtin_amdgcn_raw_buffer_store_b128(w, rO, vo, so + (unsigned)(ai * 128 + m * 16) * WP + (unsigned)bj * 256u, 0);
                }
    }
}

}

extern "C" void kernel_launch(int rank, int nranks, ncclComm_t comm,
                              hipStream_t stream,
                              void* const* d_in, const int* in_sizes, int n_in,
                              void* d_out, int out_size,
                              const int* in_dtypes, int out_dtype,
                              void* const* d_out_all,
                              void* const* d_ws_all, size_t ws_bytes) {
    (void)comm; (void)in_dtypes; (void)out_dtype; (void)d_out_all;
    if (nranks != NR || n_in < 2 || rank < 0 || rank >= NR || ws_bytes < (size_t)WS_NEED ||
        in_sizes[0] != 4096 * 512 || in_sizes[1] != 512 * 16384 || out_size != 512 * 16384) {
        fprintf(stderr, "kernel_launch: unexpected problem shape\n");
        return;
    }
    static bool attr_set[64] = {};
    int dev = 0;
    HIP_CHECK(hipGetDevice(&dev));
    if (dev >= 0 && dev < 64 && !attr_set[dev]) {
        HIP_CHECK(hipFuncSetAttribute((const void*)fused_gemm_rs, hipFuncAttributeMaxDynamicSharedMemorySize, (int)LDS_BYTES));
        attr_set[dev] = true;
    }
    Args a;
    a.x = (const char*)d_in[0];
    a.w = (const char*)d_in[1];
    a.out = (char*)d_out;
    for (int p = 0; p < NR; ++p) a.ws[p] = (char*)d_ws_all[p];
    a.rank = rank;
    hipLaunchKernelGGL(fused_gemm_rs, dim3(NBLK), dim3(512), LDS_BYTES, stream, a);
    HIP_CHECK(hipGetLastError());
}
```

```cpp
#include <hip/hip_runtime.h>
#include <rccl/rccl.h>
#include <cstdio>
#include <cstdint>

#define HIP_CHECK(expr) do { hipError_t _e = (expr); if (_e != hipSuccess) { \
    fprintf(stderr, "HIP error %s at %s:%d\n", hipGetErrorString(_e), __FILE__, __LINE__); \
    return; } } while (0)

namespace {

#define LAS __attribute__((address_space(3)))
typedef short bf16x8 __attribute__((ext_vector_type(8)));
typedef short s16x4 __attribute__((ext_vector_type(4)));
typedef float f32x4 __attribute__((ext_vector_type(4)));
typedef float f32x2 __attribute__((ext_vector_type(2)));
typedef unsigned u32x4 __attribute__((ext_vector_type(4)));
typedef __bf16 bf16x2_t __attribute__((ext_vector_type(2)));

constexpr int NR = 8;
constexpr unsigned XP = 1024;
constexpr unsigned WP = 32768;
constexpr unsigned X_BYTES = 4096u * XP;
constexpr unsigned W_BYTES = 512u * WP;
constexpr unsigned O_BYTES = 512u * WP;
constexpr unsigned UNIT = 32768;
constexpr unsigned HTB = 16384;

constexpr unsigned CTR_OFF = 0;
constexpr unsigned TMO_OFF = 16384;
constexpr unsigned QC_OFF = 20480;
constexpr unsigned XF_OFF = 32768;
constexpr unsigned WF_OFF = 36864;
constexpr unsigned PF_OFF = 49152;
constexpr unsigned STG_OFF = 1u << 20;
constexpr unsigned XS_REL = 0, WS_REL = 32u << 20, PS_REL = 48u << 20, STG_BYTES = 64u << 20;
constexpr unsigned WS_NEED = STG_OFF + 2 * STG_BYTES;

constexpr int NCOPY = 112, NGEMM = 128, NBLK = NCOPY + NGEMM;
#define OUT_AUX 0
#define WIN 1
#define NPUSH 16
#define WINP 1
#define LAGV 2
constexpr int LAG = LAGV;
constexpr int NUNITS = 176;
constexpr unsigned LDS_BYTES = 131072 + 64;
constexpr unsigned long long DEADLINE_TICKS = 40000000ull;

struct Args {
    const char* x; const char* w; char* out;
    char* ws[NR];
    int rank;
};

__device__ __forceinline__ int lds_byte(int r, int c) {
    const int st = (r >> 4) * 2 + (c >> 5), rr = r & 15, cc = c & 31, ob = rr * 64 + cc * 2;
    return st * 1024 + (ob ^ (((ob >> 9) & 1) << 5));
}
__device__ __forceinline__ void stage_rc(int b, int& R, int& C) {
    const int st = b / 1024, sb = b % 1024, swz = sb ^ (((sb >> 9) & 1) << 5);
    R = (st >> 1) * 16 + swz / 64; C = (st & 1) * 32 + (swz % 64) / 2;
}
__device__ __forceinline__ unsigned pack_bf16(float lo, float hi) {
    f32x2 v = {lo, hi};
    return __builtin_bit_cast(unsigned, __builtin_convertvector(v, bf16x2_t));
}
__device__ __forceinline__ __amdgpu_buffer_rsrc_t mk_rsrc(const void* p, unsigned bytes) {
    return __builtin_amdgcn_make_buffer_rsrc((void*)p, 0, (int)bytes, 0x00020000);
}
__device__ __forceinline__ unsigned ld_sys(const unsigned* p) { return __hip_atomic_load(p, __ATOMIC_RELAXED, __HIP_MEMORY_SCOPE_SYSTEM); }
__device__ __forceinline__ unsigned ld_agent(const unsigned* p) { return __hip_atomic_load(p, __ATOMIC_RELAXED, __HIP_MEMORY_SCOPE_AGENT); }
__device__ __forceinline__ void st_sys(unsigned* p, unsigned v) { __hip_atomic_store(p, v, __ATOMIC_RELAXED, __HIP_MEMORY_SCOPE_SYSTEM); }
__device__ __forceinline__ void st_agent(unsigned* p, unsigned v) { __hip_atomic_store(p, v, __ATOMIC_RELAXED, __HIP_MEMORY_SCOPE_AGENT); }

#define PT(i) do {} while (0)
#define PEND() do {} while (0)
#define PBEG() do {} while (0)
#define WAIT_V(n) asm volatile("s_waitcnt vmcnt(" #n ")" ::: "memory")
#define WAIT_L(n) asm volatile("s_waitcnt lgkmcnt(" #n ")" ::: "memory")
#define BAR __builtin_amdgcn_s_barrier()
#define SCHED __builtin_amdgcn_sched_barrier(0)
#define CFENCE asm volatile("" ::: "memory")

__device__ __forceinline__ void decode_unit(int n, int& kind, int& step, int& idx) {
    constexpr int P1 = 15 * LAG, P2 = P1 + 22 * (8 - LAG);
    int s, r;
    if (n < P1) { s = n / 15; r = n - s * 15; }
    else if (n < P2) { const int m = n - P1; const int q = m / 22; s = LAG + q; r = m - q * 22; }
    else { const int m = n - P2; const int q = m / 7; s = 8 + q; r = 15 + (m - q * 7); }
    if (r < 8) { kind = 0; step = s; idx = r; }
    else if (r < 15) { kind = 1; step = s; idx = r - 8; }
    else { kind = 2; step = s - LAG; idx = r - 15; }
}

__global__ __launch_bounds__(512, 2) void fused_gemm_rs(Args a) {
    extern __shared__ __attribute__((aligned(16))) unsigned char shm[];
    LAS unsigned char* lds = (LAS unsigned char*)shm;
    const int tid = threadIdx.x, wid = __builtin_amdgcn_readfirstlane(tid >> 6), lane = tid & 63;
    const int b = blockIdx.x, rank = a.rank & 7;
    char* const myws = a.ws[rank];
    unsigned* const tmo = (unsigned*)(myws + TMO_OFF);

    unsigned* const ctr = (unsigned*)(myws + CTR_OFF + (unsigned)b * 64u);
    const unsigned call = (unsigned)__builtin_amdgcn_readfirstlane((int)ld_agent(ctr)) + 1u;
    WAIT_V(0);
    __syncthreads();
    if (tid == 0) st_agent(ctr, call);
    const unsigned stg = STG_OFF + (call & 1u) * STG_BYTES;
    PT(0); PBEG();

    unsigned voffA, voffB;
    { int R, C; stage_rc(tid * 16, R, C); voffA = (unsigned)R * XP + (unsigned)C * 2u;
      const int row = tid >> 4, chp = tid & 15, f = ((row & 3) << 2) | ((row >> 2) & 3);
      voffB = (unsigned)row * WP + (unsigned)(chp ^ f) * 16u; }
    const unsigned voffS = (unsigned)tid * 16u;

    const bool is_copy = b < NCOPY;
    int mi, nj, colrank;
    if (is_copy) {
        const int dd = 1 + b % 7, c = b / 7;
        const int dest = (rank + dd) & 7;
        colrank = dest; mi = rank * 2 + (c >> 3); nj = c & 7;
        const __amdgpu_buffer_rsrc_t rD = mk_rsrc(a.ws[dest], WS_NEED);
        char* const dws = a.ws[dest];
        u32x4 r[4], rn[4];
        unsigned doff, foff;
        auto load_unit = [&](int n, u32x4 (&v)[4], unsigned& dsto, unsigned& flo) {
            int kind, step, idx; decode_unit(n, kind, step, idx);
            if (kind == 0) {
                const __amdgpu_buffer_rsrc_t rs = mk_rsrc(a.w, W_BYTES);
                const unsigned so = (unsigned)step * 64u * WP + (unsigned)(dest * 2048 + idx * 256) * 2u;
#pragma unroll
                for (int h = 0; h < 2; ++h)
#pragma unroll
                    for (int i = 0; i < 2; ++i)
                        v[h * 2 + i] = __builtin_amdgcn_raw_buffer_load_b128(rs, voffB, so + h * 256u + i * 32u * WP, 0);
                dsto = stg + WS_REL + (unsigned)((rank * 8 + step) * 8 + idx) * UNIT;
                flo = WF_OFF + (unsigned)(idx * 64 + rank * 8 + step) * 4u;
            } else {
                const int tm = ((rank + 1 + idx) & 7) * 2 + (kind - 1);
                const __amdgpu_buffer_rsrc_t rs = mk_rsrc(a.x, X_BYTES);
                const unsigned so = (unsigned)tm * 256u * XP + (unsigned)step * 128u;
#pragma unroll
                for (int h = 0; h < 2; ++h)
#pragma unroll
                    for (int i = 0; i < 2; ++i)
                        v[h * 2 + i] = __builtin_amdgcn_raw_buffer_load_b128(rs, voffA, so + h * 128u * XP + i * 64u * XP, 0);
                dsto = stg + XS_REL + (unsigned)((rank * 8 + step) * 16 + tm) * UNIT;
                flo = XF_OFF + (unsigned)(tm * 64 + rank * 8 + step) * 4u;
            }
        };
        unsigned* const qc = (unsigned*)(myws + QC_OFF + (unsigned)dd * 256u);
        const unsigned qbase = (call - 1u) * (unsigned)NUNITS - 2u * NPUSH;
        LAS volatile int* const lq = (LAS volatile int*)(lds + 131072 + 32);
        if (c < NPUSH) {
        int n1 = c + NPUSH;
        load_unit(c, r, doff, foff);
#pragma unroll 1
        for (int k = 0;; ++k) {
#pragma unroll
            for (int u = 0; u < WIN; ++u)
                __builtin_amdgcn_raw_buffer_store_b128(r[u], rD, voffS, doff + (unsigned)u * 8192u, 17);
            unsigned doffn = doff, foffn = foff;
            const bool more = n1 < NUNITS;
            if (more) load_unit(n1, rn, doffn, foffn);
            if (tid == 0) lq[k & 1] = (int)(__hip_atomic_fetch_add(qc, 1u, __ATOMIC_RELAXED, __HIP_MEMORY_SCOPE_AGENT) - qbase);
            WAIT_V(0);
#pragma unroll
            for (int u = WIN; u < 4; ++u) {
                __builtin_amdgcn_raw_buffer_store_b128(r[u], rD, voffS, doff + (unsigned)u * 8192u, 17);
                if ((u + 1) % WIN == 0) WAIT_V(0);
            }
            WAIT_L(0);
            asm volatile("" : "+v"(rn[0]), "+v"(rn[1]), "+v"(rn[2]), "+v"(rn[3]));
            __syncthreads();
            if (tid == 0) st_sys((unsigned*)(dws + foff), call);
            if (!more) break;
            n1 = __builtin_amdgcn_readfirstlane(lq[k & 1]);
#pragma unroll
            for (int u = 0; u < 4; ++u) r[u] = rn[u];
            doff = doffn; foff = foffn;
            if (k == 0) PT(1);
            if (k == 4) PT(2);
            if (k == 8) PT(3);
        }
        }
        PT(4);
    } else {
        const int g = b - NCOPY, xcd = g & 7, idx = g >> 3;
        mi = 4 * (xcd >> 1) + (idx & 3); nj = 4 * (xcd & 1) + (idx >> 2); colrank = rank;
    }
    const int rb = mi >> 1, hh = mi & 1;

    const int wr = wid >> 2, wc = wid & 3, fr = lane & 15, fq = lane >> 4;
    const unsigned ldsw = (unsigned)wid * 1024u;
    const int aoff = lds_byte(wr * 64 + fr, fq * 8);
    unsigned boff[2][2];
    { const int q = fr >> 2, p = lane & 3;
#pragma unroll
      for (int n = 0; n < 2; ++n)
#pragma unroll
          for (int t = 0; t < 2; ++t) {
              const int ch = 4 * wc + 2 * n + (p >> 1), f = (q << 2) | ((2 * fq + t) & 3);
              boff[n][t] = (unsigned)(size_t)lds + 65536u + 256u * (unsigned)(8 * fq + 4 * t + q) + 16u * (unsigned)(ch ^ f) + 8u * (unsigned)(p & 1);
          } }
#define SA(bb, h) (((bb) * 2 + (h)) * HTB)
#define SB(bb, h) ((4 + (bb) * 2 + (h)) * HTB)
#define SBR(bb, h) (((bb) * 2 + (h)) * HTB)
#define STAGE(bufoff, rs, voff, soff, istr) do { _Pragma("unroll") for (int _i = 0; _i < 2; ++_i) \
        __builtin_amdgcn_raw_ptr_buffer_load_lds(rs, (LAS void*)(lds + (bufoff) + ldsw + _i * 8192), 16, voff, (soff) + _i * (istr), 0, 16); } while (0)
#define LDA(dst, bb, h) do { _Pragma("unroll") for (int m = 0; m < 4; ++m) _Pragma("unroll") for (int k = 0; k < 2; ++k) \
        dst[m][k] = *(const LAS bf16x8*)(lds + SA(bb, h) + aoff + m * 2048 + k * 1024); } while (0)
#define TRR(d, addr, off) asm volatile("ds_read_b64_tr_b16 %0, %1 offset:%2" : "=&v"(d) : "v"(addr), "i"(off) : "memory")
#define LDB(dst, bb, h) do { _Pragma("unroll") for (int n = 0; n < 2; ++n) _Pragma("unroll") for (int k = 0; k < 2; ++k) { \
        TRR(dst[n][k][0], boff[n][0], SBR(bb, h) + k * 8192); TRR(dst[n][k][1], boff[n][1], SBR(bb, h) + k * 8192); } } while (0)
#define PKB(v) __builtin_shufflevector((v)[0], (v)[1], 0, 1, 2, 3, 4, 5, 6, 7)
#define MMA(ai, bj, At, Bt) do { SCHED; __builtin_amdgcn_s_setprio(1); _Pragma("unroll") for (int m = 0; m < 4; ++m) _Pragma("unroll") for (int n = 0; n < 2; ++n) \
        _Pragma("unroll") for (int k = 0; k < 2; ++k) acc[ai][bj][m][n] = __builtin_amdgcn_mfma_f32_16x16x32_bf16(PKB(Bt[n][k]), At[m][k], acc[ai][bj][m][n], 0, 0, 0); \
        __builtin_amdgcn_s_setprio(0); } while (0)

    f32x4 acc[2][2][4][2];
#pragma unroll
    for (int i0 = 0; i0 < 2; ++i0)
#pragma unroll
        for (int i1 = 0; i1 < 2; ++i1)
#pragma unroll
            for (int m = 0; m < 4; ++m)
#pragma unroll
                for (int n = 0; n < 2; ++n) acc[i0][i1][m][n] = (f32x4){0.f, 0.f, 0.f, 0.f};
    bf16x8 At[4][2]; s16x4 B0[2][2][2], B1[2][2][2];

    const int np = (rb == rank) ? 7 : 6;
    const unsigned long long valid = ~((0xFFull << (rank * 8)) | (0xFFull << (rb * 8)));
    const int nph = is_copy ? 1 : 2;
#pragma unroll 1
    for (int ph = 0; ph < nph; ++ph) {
        const bool own = (ph == 0);
        const int nt = own ? 8 : np * 8;
        const __amdgpu_buffer_rsrc_t rA = mk_rsrc(own ? a.x : (const char*)myws, own ? X_BYTES : WS_NEED);
        const __amdgpu_buffer_rsrc_t rB = mk_rsrc(own ? a.w : (const char*)myws, own ? W_BYTES : WS_NEED);
        const unsigned vA = own ? voffA : voffS, vB = own ? voffB : voffS;
        const unsigned iA = own ? 64u * XP : 8192u, iB = own ? 32u * WP : 8192u;
        const unsigned hA = own ? 128u * XP : HTB, hB = own ? 256u : HTB;
        auto addrA = [&](int id) -> unsigned {
            return own ? (unsigned)mi * 256u * XP + (unsigned)id * 128u
                       : stg + XS_REL + (unsigned)(id * 16 + mi) * UNIT;
        };
        auto addrB = [&](int id) -> unsigned {
            return own ? (unsigned)id * 64u * WP + (unsigned)(colrank * 2048 + nj * 256) * 2u
                       : stg + WS_REL + (unsigned)(id * 8 + nj) * UNIT;
        };
        unsigned long long ready = own ? 0xFFull : 0ull, seen = ready;
        const unsigned* const xf = (const unsigned*)(myws + XF_OFF) + mi * 64 + lane;
        const unsigned* const wf = (const unsigned*)(myws + WF_OFF) + nj * 64 + lane;
        LAS volatile unsigned* const lavail = (LAS volatile unsigned*)(lds + 131072);
        int pslot = 0;
        auto next = [&]() -> int {
            if (ready == 0ull) {
                if (wid == 0) {
                    const unsigned long long t0 = __builtin_amdgcn_s_memrealtime();
                    unsigned long long mask;
                    for (;;) {
                        const unsigned fx = ld_sys(xf), fw = ld_sys(wf);
                        const bool ok = (int)(fx - call) >= 0 && (int)(fw - call) >= 0;
                        mask = __builtin_amdgcn_ballot_w64(ok) & valid & ~seen;
                        if (mask != 0ull) break;
                        if ((unsigned)__builtin_amdgcn_readfirstlane((int)ld_agent(tmo)) == call ||
                            __builtin_amdgcn_s_memrealtime() - t0 > DEADLINE_TICKS) {
                            if (lane == 0) st_agent(tmo, call);
                            mask = valid & ~seen; break;
                        }
                        __builtin_amdgcn_s_sleep(2);
                    }
                    if (lane == 0) { lavail[pslot] = (unsigned)mask; lavail[pslot + 1] = (unsigned)(mask >> 32); }
                    WAIT_V(0); WAIT_L(0);
                }
                CFENCE; BAR; CFENCE;
                const unsigned lo = (unsigned)__builtin_amdgcn_readfirstlane((int)lavail[pslot]), hi = (unsigned)__builtin_amdgcn_readfirstlane((int)lavail[pslot + 1]);
                pslot ^= 2;
                ready = ((unsigned long long)hi << 32) | lo;
                seen |= ready;
            }
            const int id = __builtin_ctzll(ready);
            ready &= ready - 1ull;
            return id;
        };

        const int id0 = next(), id1 = next();
        { const unsigned a0 = addrA(id0), b0 = addrB(id0), a1 = addrA(id1), b1 = addrB(id1);
          STAGE(SB(0, 0), rB, vB, b0, iB); STAGE(SA(0, 0), rA, vA, a0, iA); STAGE(SB(0, 1), rB, vB, b0 + hB, iB); STAGE(SA(0, 1), rA, vA, a0 + hA, iA);
          if (wr == 1) BAR;
          WAIT_V(4); BAR;
          STAGE(SB(1, 0), rB, vB, b1, iB); STAGE(SA(1, 0), rA, vA, a1, iA); STAGE(SB(1, 1), rB, vB, b1 + hB, iB);
          WAIT_V(6); BAR; }
        unsigned a1 = addrA(id1);
#pragma unroll 1
        for (int t = 0; t < nt; t += 2) {
            const bool last = (t == nt - 2);
            const int t2 = last ? id0 : next();
            const unsigned a2 = addrA(t2), b2 = addrB(t2);
            LDB(B0, 0, 0); SCHED; LDA(At, 0, 0); STAGE(SA(1, 1), rA, vA, a1 + hA, iA);
            WAIT_L(8); BAR; WAIT_L(0); MMA(0, 0, At, B0); BAR; SCHED;
            LDB(B1, 0, 1); STAGE(SB(0, 0), rB, vB, b2, iB);
            BAR; WAIT_L(0); MMA(0, 1, At, B1); BAR;
            LDA(At, 0, 1); STAGE(SA(0, 0), rA, vA, a2, iA);
            BAR; WAIT_L(0); MMA(1, 0, At, B0); BAR; SCHED;
            STAGE(SB(0, 1), rB, vB, b2 + hB, iB);
            WAIT_V(6); BAR; MMA(1, 1, At, B1); BAR;
            const int t3 = last ? id0 : next();
            const unsigned a3 = addrA(t3), b3 = addrB(t3);
            LDB(B0, 1, 0); SCHED; LDA(At, 1, 0); STAGE(SA(0, 1), rA, vA, a2 + hA, iA);
            WAIT_L(8); BAR; WAIT_L(0); MMA(0, 0, At, B0); BAR; SCHED;
            LDB(B1, 1, 1); STAGE(SB(1, 0), rB, vB, b3, iB);
            BAR; WAIT_L(0); MMA(0, 1, At, B1); BAR;
            LDA(At, 1, 1); STAGE(SA(1, 0), rA, vA, a3, iA);
            BAR; WAIT_L(0); MMA(1, 0, At, B0); BAR; SCHED;
            STAGE(SB(1, 1), rB, vB, b3 + hB, iB);
            WAIT_V(6); BAR; MMA(1, 1, At, B1); BAR;
            a1 = a3;
        }
        WAIT_V(0);
        if (wr == 0) BAR;
        BAR;
        PT(own ? (is_copy ? 5 : 1) : 9);
    }

    if (!is_copy && rb != rank) {
        const bool bad = (unsigned)__builtin_amdgcn_readfirstlane((int)ld_agent(tmo)) == call;
        const __amdgpu_buffer_rsrc_t rP = mk_rsrc(a.ws[rb], WS_NEED);
        const unsigned slot = (unsigned)((rank * 2 + hh) * 8 + nj);
        const unsigned base = stg + PS_REL + slot * 131072u + (unsigned)wid * 16384u;
#pragma unroll
        for (int ai = 0; ai < 2; ++ai)
#pragma unroll
            for (int bj = 0; bj < 2; ++bj)
#pragma unroll
                for (int m = 0; m < 4; ++m) {
                    const f32x4 v0 = acc[ai][bj][m][0], v1 = acc[ai][bj][m][1];
                    u32x4 w; w.x = pack_bf16(v0[0], v0[1]); w.y = pack_bf16(v0[2], v0[3]); w.z = pack_bf16(v1[0], v1[1]); w.w = pack_bf16(v1[2], v1[3]);
                    if (bad) w = (u32x4){0x7FC07FC0u, 0x7FC07FC0u, 0x7FC07FC0u, 0x7FC07FC0u};
                    __builtin_amdgcn_raw_buffer_store_b128(w, rP, (unsigned)lane * 16u, base + (unsigned)((ai * 2 + bj) * 4 + m) * 1024u, 17);
                    if ((m + 1) % WINP == 0 && WINP <= 4) WAIT_V(0);
                }
        WAIT_V(0);
        if (lane == 0) st_sys((unsigned*)(a.ws[rb] + PF_OFF) + slot * 32u + wid, call);
        PT(10); PEND();
        return;
    }
    if (is_copy) {
        const unsigned slot = (unsigned)((colrank * 2 + hh) * 8 + nj);
        const unsigned* const pf = (const unsigned*)(myws + PF_OFF) + slot * 32u + wid;
        const unsigned long long t0 = __builtin_amdgcn_s_memrealtime();
        while ((int)((unsigned)__builtin_amdgcn_readfirstlane((int)ld_sys(pf)) - call) < 0) {
            if ((unsigned)__builtin_amdgcn_readfirstlane((int)ld_agent(tmo)) == call ||
                __builtin_amdgcn_s_memrealtime() - t0 > DEADLINE_TICKS) {
                if (lane == 0) st_agent(tmo, call);
                break;
            }
            __builtin_amdgcn_s_sleep(2);
        }
        CFENCE;
        PT(6);
        const __amdgpu_buffer_rsrc_t rP = mk_rsrc(myws, WS_NEED);
        const unsigned base = stg + PS_REL + slot * 131072u + (unsigned)wid * 16384u;
        u32x4 pv[16];
#pragma unroll
        for (int u = 0; u < 16; ++u) pv[u] = __builtin_amdgcn_raw_buffer_load_b128(rP, (unsigned)lane * 16u, base + (unsigned)u * 1024u, 17);
#pragma unroll
        for (int ai = 0; ai < 2; ++ai)
#pragma unroll
            for (int bj = 0; bj < 2; ++bj)
#pragma unroll
                for (int m = 0; m < 4; ++m) {
                    const u32x4 w = pv[(ai * 2 + bj) * 4 + m];
                    f32x4& v0 = acc[ai][bj][m][0]; f32x4& v1 = acc[ai][bj][m][1];
                    v0[0] += __uint_as_float(w.x << 16); v0[1] += __uint_as_float(w.x & 0xFFFF0000u);
                    v0[2] += __uint_as_float(w.y << 16); v0[3] += __uint_as_float(w.y & 0xFFFF0000u);
                    v1[0] += __uint_as_float(w.z << 16); v1[1] += __uint_as_float(w.z & 0xFFFF0000u);
                    v1[2] += __uint_as_float(w.w << 16); v1[3] += __uint_as_float(w.w & 0xFFFF0000u);
                }
    }
    {
        const bool bad = (unsigned)__builtin_amdgcn_readfirstlane((int)ld_agent(tmo)) == call;
        const __amdgpu_buffer_rsrc_t rO = mk_rsrc(a.out, O_BYTES);
        const unsigned vo = (unsigned)(wr * 64 + fr) * WP + (unsigned)(wc * 32 + ((fq & 1) << 4) + ((fq >> 1) << 3)) * 2u;
        const unsigned so = (unsigned)hh * 256u * WP + (unsigned)(colrank * 2048 + nj * 256) * 2u;
#pragma unroll
        for (int ai = 0; ai < 2; ++ai)
#pragma unroll
            for (int m = 0; m < 4; ++m)
#pragma unroll
                for (int bj = 0; bj < 2; ++bj) {
                    const f32x4 v0 = acc[ai][bj][m][0], v1 = acc[ai][bj][m][1];
                    const unsigned ax = pack_bf16(v0[0], v0[1]), ay = pack_bf16(v0[2], v0[3]);
                    const unsigned bx = pack_bf16(v1[0], v1[1]), by = pack_bf16(v1[2], v1[3]);
                    const auto sx = __builtin_amdgcn_permlane16_swap(ax, bx, false, false);
                    const auto sy = __builtin_amdgcn_permlane16_swap(ay, by, false, false);
                    u32x4 w = {sx[0], sy[0], sx[1], sy[1]};
                    if (bad) w = (u32x4){0x7FC07FC0u, 0x7FC07FC0u, 0x7FC07FC0u, 0x7FC07FC0u};
                    __builtin_amdgcn_raw_buffer_store_b128(w, rO, vo, so + (unsigned)(ai * 128 + m * 16) * WP + (unsigned)bj * 256u, OUT_AUX);
                }
    }
}

}

extern "C" void kernel_launch(int rank, int nranks, ncclComm_t comm,
                              hipStream_t stream,
                              void* const* d_in, const int* in_sizes, int n_in,
                              void* d_out, int out_size,
                              const int* in_dtypes, int out_dtype,
                              void* const* d_out_all,
                              void* const* d_ws_all, size_t ws_bytes) {
    (void)comm; (void)in_dtypes; (void)out_dtype; (void)d_out_all;
    if (nranks != NR || n_in < 2 || rank < 0 || rank >= NR || ws_bytes < (size_t)WS_NEED ||
        in_sizes[0] != 4096 * 512 || in_sizes[1] != 512 * 16384 || out_size != 512 * 16384) {
        fprintf(stderr, "kernel_launch: unexpected problem shape\n");
        return;
    }
    static bool attr_set[64] = {};
    int dev = 0;
    HIP_CHECK(hipGetDevice(&dev));
    if (dev >= 0 && dev < 64 && !attr_set[dev]) {
        HIP_CHECK(hipFuncSetAttribute((const void*)fused_gemm_rs, hipFuncAttributeMaxDynamicSharedMemorySize, (int)LDS_BYTES));
        attr_set[dev] = true;
    }
    Args a;
    a.x = (const char*)d_in[0];
    a.w = (const char*)d_in[1];
    a.out = (char*)d_out;
    for (int p = 0; p < NR; ++p) a.ws[p] = (char*)d_ws_all[p];
    a.rank = rank;
    hipLaunchKernelGGL(fused_gemm_rs, dim3(NBLK), dim3(512), LDS_BYTES, stream, a);
    HIP_CHECK(hipGetLastError());
}
```

```cpp
#include <hip/hip_runtime.h>
#include <rccl/rccl.h>
#include <cstdio>
#include <cstdint>

#define HIP_CHECK(expr) do { hipError_t _e = (expr); if (_e != hipSuccess) { \
    fprintf(stderr, "HIP error %s at %s:%d\n", hipGetErrorString(_e), __FILE__, __LINE__); \
    return; } } while (0)

namespace {

#define LAS __attribute__((address_space(3)))
typedef short bf16x8 __attribute__((ext_vector_type(8)));
typedef short s16x4 __attribute__((ext_vector_type(4)));
typedef short s16x8 __attribute__((ext_vector_type(8)));
typedef float f32x4 __attribute__((ext_vector_type(4)));
typedef float f32x2 __attribute__((ext_vector_type(2)));
typedef unsigned u32x4 __attribute__((ext_vector_type(4)));
typedef __bf16 bf16x2_t __attribute__((ext_vector_type(2)));

constexpr int NR = 8;
constexpr unsigned XP = 1024;
constexpr unsigned WP = 32768;
constexpr unsigned X_BYTES = 4096u * XP;
constexpr unsigned W_BYTES = 512u * WP;
constexpr unsigned O_BYTES = 512u * WP;
constexpr unsigned UNIT = 32768;
constexpr unsigned HTB = 16384;

constexpr unsigned CTR_OFF = 0;
constexpr unsigned TMO_OFF = 16384;
constexpr unsigned QC_OFF = 20480;
constexpr unsigned SF_OFF = 24576;
constexpr unsigned PF_OFF = 32768;
constexpr unsigned XF_OFF = 65536;
constexpr unsigned WF_OFF = 81920;
constexpr unsigned XD_OFF = 1u << 20;
constexpr unsigned WD_OFF = 33u << 20, WD_BYTES = 16u << 20;
constexpr unsigned SR_OFF = 65u << 20, SR_BYTES = 18u << 20;
constexpr unsigned PS_OFF = 101u << 20, PS_BYTES = 16u << 20;
constexpr unsigned WS_NEED = PS_OFF + 2 * PS_BYTES;

constexpr int NCOPY = 112, NGEMM = 128, NBLK = NCOPY + NGEMM;
#define OUT_AUX 17
#define WINU 2
#define WINL 2
#define ROT 1
constexpr int NUNITS = 176;
constexpr int NPUSHED = 8;
constexpr int NPULL = NUNITS - NPUSHED;
constexpr int NITEMS = 504;
constexpr unsigned LDS_BYTES = 131072 + 64;
constexpr unsigned long long DEADLINE_TICKS = 40000000ull;

struct Args {
    const char* x; const char* w; char* out;
    char* ws[NR];
    int rank;
};

__device__ __forceinline__ int lds_byte(int r, int c) {
    const int st = (r >> 4) * 2 + (c >> 5), rr = r & 15, cc = c & 31, ob = rr * 64 + cc * 2;
    return st * 1024 + (ob ^ (((ob >> 9) & 1) << 5));
}
__device__ __forceinline__ void stage_rc(int b, int& R, int& C) {
    const int st = b / 1024, sb = b % 1024, swz = sb ^ (((sb >> 9) & 1) << 5);
    R = (st >> 1) * 16 + swz / 64; C = (st & 1) * 32 + (swz % 64) / 2;
}
__device__ __forceinline__ unsigned pack_bf16(float lo, float hi) {
    f32x2 v = {lo, hi};
    return __builtin_bit_cast(unsigned, __builtin_convertvector(v, bf16x2_t));
}
__device__ __forceinline__ __amdgpu_buffer_rsrc_t mk_rsrc(const void* p, unsigned bytes) {
    return __builtin_amdgcn_make_buffer_rsrc((void*)p, 0, (int)bytes, 0x00020000);
}
template <int AUX>
__device__ __forceinline__ void store16(const u32x4 w, const __amdgpu_buffer_rsrc_t rs, unsigned voff, unsigned soff) {
    __builtin_amdgcn_raw_buffer_store_b128(w, rs, voff, soff, AUX);
    asm volatile("s_nop 1" :: "v"(w) : "memory");
}
__device__ __forceinline__ unsigned ld_sys(const unsigned* p) { return __hip_atomic_load(p, __ATOMIC_RELAXED, __HIP_MEMORY_SCOPE_SYSTEM); }
__device__ __forceinline__ unsigned ld_agent(const unsigned* p) { return __hip_atomic_load(p, __ATOMIC_RELAXED, __HIP_MEMORY_SCOPE_AGENT); }
__device__ __forceinline__ void st_sys(unsigned* p, unsigned v) { __hip_atomic_store(p, v, __ATOMIC_RELAXED, __HIP_MEMORY_SCOPE_SYSTEM); }
__device__ __forceinline__ void st_agent(unsigned* p, unsigned v) { __hip_atomic_store(p, v, __ATOMIC_RELAXED, __HIP_MEMORY_SCOPE_AGENT); }

#define PT(i) do {} while (0)
#define PEND() do {} while (0)
#define PBEG() do {} while (0)
#define WAIT_V(n) asm volatile("s_waitcnt vmcnt(" #n ")" ::: "memory")
#define WAIT_L(n) asm volatile("s_waitcnt lgkmcnt(" #n ")" ::: "memory")
#define BAR __builtin_amdgcn_s_barrier()
#define SCHED __builtin_amdgcn_sched_barrier(0)
#define CFENCE asm volatile("" ::: "memory")

#define SEQ0 0x1202091101100800ull
#define SEQ1 0x14040b1913030a18ull
#define SEQ2 0x16060d1b15050c1aull
#define SEQ3 0x1f1e0f1d0e17071cull
__device__ __forceinline__ int group_code(int g) {
    const unsigned long long q = g < 8 ? SEQ0 : g < 16 ? SEQ1 : g < 24 ? SEQ2 : SEQ3;
    return (int)(q >> ((g & 7) * 8)) & 31;
}
struct Cursor { int g, pos, item; };
__device__ __forceinline__ void decode_unit(Cursor& cu, int n, int dl, int& kind, int& step, int& idx) {
    int code = group_code(cu.g);
    while (n >= cu.pos + (code < 16 ? 4 : 7)) { cu.pos += code < 16 ? 4 : 7; ++cu.g; code = group_code(cu.g); }
    const int r = n - cu.pos;
    step = code & 7;
    if (code < 16) { kind = 0; idx = (code >> 3) * 4 + ((r + ROT * dl) & 3); }
    else { kind = (code >> 3) - 1; idx = r; }
}
__device__ __forceinline__ void decode_item(Cursor& cu, int t, int& n, int& dl) {
    int code = group_code(cu.g);
    for (;;) {
        const int cnt = cu.g < 2 ? 0 : code < 16 ? 28 : 7;
        if (t < cu.item + cnt) break;
        cu.item += cnt; cu.pos += code < 16 ? 4 : 7; ++cu.g; code = group_code(cu.g);
    }
    const int r = t - cu.item;
    if (code < 16) { const int q = r / 7; n = cu.pos + q; dl = 1 + r - q * 7; }
    else { n = cu.pos + r; dl = 0; }
}
__device__ __forceinline__ unsigned src_slot(int kind, int step, int idx, int dl) {
    return kind == 0 ? (unsigned)((dl - 1) * 64 + step * 8 + idx) : (unsigned)(448 + (kind - 1) * 56 + step * 7 + idx);
}

__global__ __launch_bounds__(512, 2) void fused_gemm_rs(Args a) {
    extern __shared__ __attribute__((aligned(16))) unsigned char shm[];
    LAS unsigned char* lds = (LAS unsigned char*)shm;
    const int tid = threadIdx.x, wid = __builtin_amdgcn_readfirstlane(tid >> 6), lane = tid & 63;
    const int b = blockIdx.x, rank = a.rank & 7;
    char* const myws = a.ws[rank];
    unsigned* const tmo = (unsigned*)(myws + TMO_OFF);

    unsigned voffA, voffB;
    { int R, C; stage_rc(tid * 16, R, C); voffA = (unsigned)R * XP + (unsigned)C * 2u;
      const int row = tid >> 4, chp = tid & 15, f = ((row & 3) << 2) | ((row >> 2) & 3);
      voffB = (unsigned)row * WP + (unsigned)(chp ^ f) * 16u; }
    const unsigned voffS = (unsigned)tid * 16u;

    const bool is_copy = b < NCOPY;
    const int dd = 1 + b % 7, c = b / 7;
    const int peer = (rank + dd) & 7;
    u32x4 r[4], rn[4];
    auto load_unit = [&](int kind, int step, int idx, int dest, u32x4 (&v)[4]) {
        if (kind == 0) {
            const __amdgpu_buffer_rsrc_t rs = mk_rsrc(a.w, W_BYTES);
            const unsigned so = (unsigned)step * 64u * WP + (unsigned)(dest * 2048 + idx * 256) * 2u;
#pragma unroll
            for (int h = 0; h < 2; ++h)
#pragma unroll
                for (int i = 0; i < 2; ++i)
                    v[h * 2 + i] = __builtin_amdgcn_raw_buffer_load_b128(rs, voffB, so + h * 256u + i * 32u * WP, 0);
        } else {
            const int tm = ((rank + 1 + idx) & 7) * 2 + (kind - 1);
            const __amdgpu_buffer_rsrc_t rs = mk_rsrc(a.x, X_BYTES);
            const unsigned so = (unsigned)tm * 256u * XP + (unsigned)step * 128u;
#pragma unroll
            for (int h = 0; h < 2; ++h)
#pragma unroll
                for (int i = 0; i < 2; ++i)
                    v[h * 2 + i] = __builtin_amdgcn_raw_buffer_load_b128(rs, voffA, so + h * 128u * XP + i * 64u * XP, 0);
        }
    };
    int in = 0, idl = 0, pu = 0;
    Cursor cu = {0, 0, 0};
    if (is_copy) {
        int kind, step, idx;
        if (c < 8) { decode_item(cu, b, in, idl); decode_unit(cu, in, idl, kind, step, idx); load_unit(kind, step, idx, (rank + idl) & 7, r); }
        else { decode_unit(cu, c - 8, dd, kind, step, idx); load_unit(0, step, idx, peer, r); pu = step * 8 + idx; }
    }

    unsigned* const ctr = (unsigned*)(myws + CTR_OFF + (unsigned)b * 64u);
    const unsigned call = (unsigned)__builtin_amdgcn_readfirstlane((int)ld_agent(ctr)) + 1u;
    WAIT_V(0);
    __syncthreads();
    if (tid == 0) st_agent(ctr, call);
    const unsigned par = call & 1u;
    PT(0); PBEG();
    const __amdgpu_buffer_rsrc_t rM = mk_rsrc(myws, WS_NEED);

    int mi, nj, colrank;
    if (is_copy) {
        colrank = peer; mi = rank * 2 + (c >> 3); nj = c & 7;
        if (c < 8) {
#pragma unroll 1
            for (int k = 0;; ++k) {
                int kind, step, idx; decode_unit(cu, in, idl, kind, step, idx);
                const unsigned doff = SR_OFF + par * SR_BYTES + src_slot(kind, step, idx, idl) * UNIT;
#pragma unroll
                for (int u = 0; u < 4; ++u)
                    store16<17>(r[u], rM, voffS, doff + (unsigned)u * 8192u);
                const int fn = in, fdl = idl;
                const bool more = k + 1 < NITEMS / 56;
                if (more) { decode_item(cu, b + (k + 1) * 56, in, idl); int k2, s2, i2; decode_unit(cu, in, idl, k2, s2, i2); load_unit(k2, s2, i2, (rank + idl) & 7, rn); }
                WAIT_V(0);
                asm volatile("" : "+v"(rn[0]), "+v"(rn[1]), "+v"(rn[2]), "+v"(rn[3]));
                __syncthreads();
                if (tid == 0) {
                    const unsigned fo = SF_OFF + (unsigned)(rank * 192 + fn) * 4u;
                    if (fdl) st_sys((unsigned*)(a.ws[(rank + fdl) & 7] + fo), call);
                    else {
#pragma unroll
                        for (int p = 1; p < 8; ++p) st_sys((unsigned*)(a.ws[(rank + p) & 7] + fo), call);
                    }
                }
                if (k == 0) PT(1);
                if (!more) break;
#pragma unroll
                for (int u = 0; u < 4; ++u) r[u] = rn[u];
            }
        } else {
            const int cp = c - 8, dl = 8 - dd;
            const __amdgpu_buffer_rsrc_t rP = mk_rsrc(a.ws[peer], WS_NEED);
            unsigned qoff = QC_OFF + (unsigned)(dd * 8 + wid) * 64u;
            asm volatile("" : "+v"(qoff));
            unsigned* const qc = (unsigned*)(myws + qoff);
            const unsigned qbase = (call - 1u) * (unsigned)NPULL - 8u;
            const unsigned* const sf = (const unsigned*)(myws + SF_OFF) + peer * 192 + lane;
            int su = NPUSHED;
            unsigned f0, f1, f2;
            auto sf_issue = [&]() { f0 = ld_sys(sf); f1 = ld_sys(sf + 64); f2 = ld_sys(sf + 128); };
            auto sf_eval = [&]() {
                const unsigned long long m0 = ~__builtin_amdgcn_ballot_w64((int)(f0 - call) >= 0 || lane < NPUSHED);
                const unsigned long long m1 = ~__builtin_amdgcn_ballot_w64((int)(f1 - call) >= 0);
                const unsigned long long m2 = ~__builtin_amdgcn_ballot_w64((int)(f2 - call) >= 0 || lane >= NUNITS - 128);
                su = m0 ? __builtin_ctzll(m0) : m1 ? 64 + __builtin_ctzll(m1) : m2 ? 128 + __builtin_ctzll(m2) : 192;
            };
            auto sf_wait = [&](int n) {
                if (n < su) return;
                const unsigned long long t0 = __builtin_amdgcn_s_memrealtime();
                for (;;) {
                    sf_issue(); sf_eval();
                    if (n < su) break;
                    if ((unsigned)__builtin_amdgcn_readfirstlane((int)ld_agent(tmo)) == call ||
                        __builtin_amdgcn_s_memrealtime() - t0 > DEADLINE_TICKS) {
                        if (lane == 0) st_agent(tmo, call);
                        su = 192; break;
                    }
                    __builtin_amdgcn_s_sleep(2);
                }
            };
            auto unit = [&](int q, int& n, unsigned& so, unsigned& dsto, unsigned& flo) {
                n = NPUSHED + q;
                int kind, step, idx; decode_unit(cu, n, dl, kind, step, idx);
                so = SR_OFF + par * SR_BYTES + src_slot(kind, step, idx, dl) * UNIT;
                if (kind == 0) {
                    dsto = WD_OFF + par * WD_BYTES + (unsigned)((peer * 8 + step) * 8 + idx) * UNIT;
                    flo = WF_OFF + (unsigned)(idx * 64 + peer * 8 + step) * 16u + (unsigned)wid * 2u;
                } else {
                    const int tm = ((peer + 1 + idx) & 7) * 2 + (kind - 1);
                    dsto = XD_OFF + (unsigned)((peer * 8 + step) * 16 + tm) * UNIT;
                    flo = XF_OFF + (unsigned)(tm * 64 + peer * 8 + step) * 16u + (unsigned)wid * 2u;
                }
            };
            auto set_flag = [&](const __amdgpu_buffer_rsrc_t rs, unsigned flo) {
                if (lane == 0) __builtin_amdgcn_raw_buffer_store_b16((unsigned short)call, rs, 0, flo, 17);
            };
            const unsigned poff = WD_OFF + par * WD_BYTES + (unsigned)(rank * 64 + pu) * UNIT;
            store16<17>(r[0], rP, voffS, poff); WAIT_V(0);
            store16<17>(r[1], rP, voffS, poff + 8192u); WAIT_V(0);
            store16<17>(r[2], rP, voffS, poff + 16384u); sf_issue(); WAIT_V(0);
            store16<17>(r[3], rP, voffS, poff + 24576u);
            sf_eval();
            int n0, n1 = 0; unsigned so0, do0, fl0, so1 = 0, do1 = 0, fl1 = 0, flp = 0;
            unit(cp, n0, so0, do0, fl0);
            sf_wait(n0);
            u32x4 v = __builtin_amdgcn_raw_buffer_load_b128(rP, voffS, so0, 17);
            bool pend = false;
            unsigned qv = 0;
#pragma unroll 1
            for (int k = 0;; ++k) {
                bool more = false;
#pragma unroll
                for (int u = 0; u < 4; ++u) {
                    WAIT_V(0);
                    asm volatile("" : "+v"(v));
                    const u32x4 w = v;
                    if (u < 3) v = __builtin_amdgcn_raw_buffer_load_b128(rP, voffS, so0 + (unsigned)(u + 1) * 8192u, 17);
                    else if (more) {
                        if (pend) { sf_eval(); pend = false; }
                        sf_wait(n1);
                        v = __builtin_amdgcn_raw_buffer_load_b128(rP, voffS, so1, 17);
                        if (su < NUNITS) { sf_issue(); pend = true; }
                    }
                    store16<17>(w, rM, voffS, do0 + (unsigned)u * 8192u);
                    if (u == 0) {
                        if (k == 0) { set_flag(rP, WF_OFF + (unsigned)((pu & 7) * 64 + rank * 8 + (pu >> 3)) * 16u + (unsigned)wid * 2u); PT(1); }
                        else set_flag(rM, flp);
                        if (lane == 0) qv = __hip_atomic_fetch_add(qc, 1u, __ATOMIC_RELAXED, __HIP_MEMORY_SCOPE_AGENT);
                    }
                    if (u == 2) {
                        const int qn = __builtin_amdgcn_readfirstlane((int)(qv - qbase));
                        more = qn < NPULL;
                        if (more) unit(qn, n1, so1, do1, fl1);
                    }
                }
                flp = fl0;
                if (!more) break;
                so0 = so1; do0 = do1; fl0 = fl1;
                if (k == 4) PT(2);
                if (k == 10) PT(3);
            }
            WAIT_V(0);
            set_flag(rM, flp);
        }
        PT(4);
    } else {
        const int g = b - NCOPY, xcd = g & 7, idx = g >> 3;
        mi = 4 * (xcd >> 1) + (idx & 3); nj = 4 * (xcd & 1) + (idx >> 2); colrank = rank;
    }
    const int rb = mi >> 1, hh = mi & 1;

    const int wr = wid >> 2, wc = wid & 3, fr = lane & 15, fq = lane >> 4;
    const unsigned ldsw = (unsigned)wid * 1024u;
    const int aoff = lds_byte(wr * 64 + fr, fq * 8);
    unsigned boff[2][2];
    { const int q = fr >> 2, p = lane & 3;
#pragma unroll
      for (int n = 0; n < 2; ++n)
#pragma unroll
          for (int t = 0; t < 2; ++t) {
              const int ch = 4 * wc + 2 * n + (p >> 1), f = (q << 2) | ((2 * fq + t) & 3);
              boff[n][t] = (unsigned)(size_t)lds + 65536u + 256u * (unsigned)(8 * fq + 4 * t + q) + 16u * (unsigned)(ch ^ f) + 8u * (unsigned)(p & 1);
          } }
#define SA(bb, h) (((bb) * 2 + (h)) * HTB)
#define SB(bb, h) ((4 + (bb) * 2 + (h)) * HTB)
#define SBR(bb, h) (((bb) * 2 + (h)) * HTB)
#define STAGE(bufoff, rs, voff, soff, istr) do { _Pragma("unroll") for (int _i = 0; _i < 2; ++_i) \
        __builtin_amdgcn_raw_ptr_buffer_load_lds(rs, (LAS void*)(lds + (bufoff) + ldsw + _i * 8192), 16, voff, (soff) + _i * (istr), 0, 16); } while (0)
#define LDA(dst, bb, h) do { _Pragma("unroll") for (int m = 0; m < 4; ++m) _Pragma("unroll") for (int k = 0; k < 2; ++k) \
        dst[m][k] = *(const LAS bf16x8*)(lds + SA(bb, h) + aoff + m * 2048 + k * 1024); } while (0)
#define TRR(d, addr, off) asm volatile("ds_read_b64_tr_b16 %0, %1 offset:%2" : "=&v"(d) : "v"(addr), "i"(off) : "memory")
#define LDB(dst, bb, h) do { _Pragma("unroll") for (int n = 0; n < 2; ++n) _Pragma("unroll") for (int k = 0; k < 2; ++k) { \
        TRR(dst[n][k][0], boff[n][0], SBR(bb, h) + k * 8192); TRR(dst[n][k][1], boff[n][1], SBR(bb, h) + k * 8192); } } while (0)
#define PKB(v) __builtin_shufflevector((v)[0], (v)[1], 0, 1, 2, 3, 4, 5, 6, 7)
#define MMA(ai, bj, At, Bt) do { SCHED; __builtin_amdgcn_s_setprio(1); _Pragma("unroll") for (int m = 0; m < 4; ++m) _Pragma("unroll") for (int n = 0; n < 2; ++n) \
        _Pragma("unroll") for (int k = 0; k < 2; ++k) acc[ai][bj][m][n] = __builtin_amdgcn_mfma_f32_16x16x32_bf16(PKB(Bt[n][k]), At[m][k], acc[ai][bj][m][n], 0, 0, 0); \
        __builtin_amdgcn_s_setprio(0); } while (0)

    f32x4 acc[2][2][4][2];
#pragma unroll
    for (int i0 = 0; i0 < 2; ++i0)
#pragma unroll
        for (int i1 = 0; i1 < 2; ++i1)
#pragma unroll
            for (int m = 0; m < 4; ++m)
#pragma unroll
                for (int n = 0; n < 2; ++n) acc[i0][i1][m][n] = (f32x4){0.f, 0.f, 0.f, 0.f};
    bf16x8 At[4][2]; s16x4 B0[2][2][2], B1[2][2][2];

    const int np = (rb == rank) ? 7 : 6;
    const unsigned long long valid = ~((0xFFull << (rank * 8)) | (0xFFull << (rb * 8)));
    const int nph = is_copy ? 1 : 2;
#pragma unroll 1
    for (int ph = 0; ph < nph; ++ph) {
        const bool own = (ph == 0);
        const int nt = own ? 8 : np * 8;
        const __amdgpu_buffer_rsrc_t rA = mk_rsrc(own ? a.x : (const char*)myws, own ? X_BYTES : WS_NEED);
        const __amdgpu_buffer_rsrc_t rB = mk_rsrc(own ? a.w : (const char*)myws, own ? W_BYTES : WS_NEED);
        const unsigned vA = own ? voffA : voffS, vB = own ? voffB : voffS;
        const unsigned iA = own ? 64u * XP : 8192u, iB = own ? 32u * WP : 8192u;
        const unsigned hA = own ? 128u * XP : HTB, hB = own ? 256u : HTB;
        auto addrA = [&](int id) -> unsigned {
            return own ? (unsigned)mi * 256u * XP + (unsigned)id * 128u
                       : XD_OFF + (unsigned)(id * 16 + mi) * UNIT;
        };
        auto addrB = [&](int id) -> unsigned {
            return own ? (unsigned)id * 64u * WP + (unsigned)(colrank * 2048 + nj * 256) * 2u
                       : WD_OFF + par * WD_BYTES + (unsigned)(id * 8 + nj) * UNIT;
        };
        unsigned long long ready = own ? 0xFFull : 0ull, seen = ready;
        const unsigned xf = XF_OFF + (unsigned)(mi * 64) * 16u, wf = WF_OFF + (unsigned)(nj * 64) * 16u;
        LAS volatile unsigned* const lavail = (LAS volatile unsigned*)(lds + 131072);
        int pslot = 0;
        const short c16 = (short)call;
        const s16x8 c8 = {c16, c16, c16, c16, c16, c16, c16, c16};
        auto arrived = [&](const u32x4 fx, const u32x4 fw) -> unsigned long long {
            const s16x8 dx = __builtin_bit_cast(s16x8, fx) - c8, dw = __builtin_bit_cast(s16x8, fw) - c8;
            const u32x4 dd4 = __builtin_bit_cast(u32x4, dx) | __builtin_bit_cast(u32x4, dw);
            const bool ok = ((dd4.x | dd4.y | dd4.z | dd4.w) & 0x80008000u) == 0u;
            return __builtin_amdgcn_ballot_w64(ok) & valid & ~seen;
        };
        auto pop = [&]() -> int { const int id = __builtin_ctzll(ready); ready &= ready - 1ull; return id; };
        int remaining = nt;
#pragma unroll 1
        while (remaining > 0) {
#pragma unroll 1
            while (__builtin_popcountll(ready) < 2) {
                if (wid == 0) {
                    const unsigned long long t0 = __builtin_amdgcn_s_memrealtime();
                    unsigned long long mask;
                    for (;;) {
                        CFENCE;
                        const u32x4 fx = __builtin_amdgcn_raw_buffer_load_b128(rM, (unsigned)lane * 16u, xf, 17);
                        const u32x4 fw = __builtin_amdgcn_raw_buffer_load_b128(rM, (unsigned)lane * 16u, wf, 17);
                        mask = arrived(fx, fw);
                        if (mask != 0ull) break;
                        if ((unsigned)__builtin_amdgcn_readfirstlane((int)ld_agent(tmo)) == call ||
                            __builtin_amdgcn_s_memrealtime() - t0 > DEADLINE_TICKS) {
                            if (lane == 0) st_agent(tmo, call);
                            mask = valid & ~seen; break;
                        }
                        __builtin_amdgcn_s_sleep(2);
                    }
                    if (lane == 0) { lavail[pslot] = (unsigned)mask; lavail[pslot + 1] = (unsigned)(mask >> 32); }
                    WAIT_V(0); WAIT_L(0);
                }
                CFENCE; BAR; CFENCE;
                const unsigned lo = (unsigned)__builtin_amdgcn_readfirstlane((int)lavail[pslot]), hi = (unsigned)__builtin_amdgcn_readfirstlane((int)lavail[pslot + 1]);
                pslot ^= 2;
                const unsigned long long m = ((unsigned long long)hi << 32) | lo;
                ready |= m; seen |= m;
            }
            const int id0 = pop(), id1 = pop();
            { const unsigned a0 = addrA(id0), b0 = addrB(id0), a1 = addrA(id1), b1 = addrB(id1);
              STAGE(SB(0, 0), rB, vB, b0, iB); STAGE(SA(0, 0), rA, vA, a0, iA); STAGE(SB(0, 1), rB, vB, b0 + hB, iB); STAGE(SA(0, 1), rA, vA, a0 + hA, iA);
              if (wr == 1) BAR;
              WAIT_V(4); BAR;
              STAGE(SB(1, 0), rB, vB, b1, iB); STAGE(SA(1, 0), rA, vA, a1, iA); STAGE(SB(1, 1), rB, vB, b1 + hB, iB);
              WAIT_V(6); BAR; }
            unsigned a1 = addrA(id1);
#pragma unroll 1
            for (;;) {
                remaining -= 2;
                const bool last = __builtin_popcountll(ready) < 2;
                const int t2 = last ? id0 : pop();
                const int t3 = last ? id0 : pop();
                const bool look = !own && (valid & ~seen) != 0ull;
                const unsigned a2 = addrA(t2), b2 = addrB(t2);
                const unsigned a3 = addrA(t3), b3 = addrB(t3);
                u32x4 fx, fw;
                LDB(B0, 0, 0); SCHED; LDA(At, 0, 0); STAGE(SA(1, 1), rA, vA, a1 + hA, iA);
                WAIT_L(8); BAR; WAIT_L(0); MMA(0, 0, At, B0); BAR; SCHED;
                LDB(B1, 0, 1); STAGE(SB(0, 0), rB, vB, b2, iB);
                BAR; WAIT_L(0); MMA(0, 1, At, B1); BAR;
                LDA(At, 0, 1); STAGE(SA(0, 0), rA, vA, a2, iA);
                BAR; WAIT_L(0); MMA(1, 0, At, B0); BAR; SCHED;
                STAGE(SB(0, 1), rB, vB, b2 + hB, iB);
                WAIT_V(6); BAR; MMA(1, 1, At, B1); BAR;
                if (look && wid == 0) {
                    fx = __builtin_amdgcn_raw_buffer_load_b128(rM, (unsigned)lane * 16u, xf, 17);
                    fw = __builtin_amdgcn_raw_buffer_load_b128(rM, (unsigned)lane * 16u, wf, 17);
                }
                LDB(B0, 1, 0); SCHED; LDA(At, 1, 0); STAGE(SA(0, 1), rA, vA, a2 + hA, iA);
                WAIT_L(8); BAR; WAIT_L(0); MMA(0, 0, At, B0); BAR; SCHED;
                LDB(B1, 1, 1); STAGE(SB(1, 0), rB, vB, b3, iB);
                BAR; WAIT_L(0); MMA(0, 1, At, B1); BAR;
                LDA(At, 1, 1); STAGE(SA(1, 0), rA, vA, a3, iA);
                BAR; WAIT_L(0); MMA(1, 0, At, B0); BAR; SCHED;
                STAGE(SB(1, 1), rB, vB, b3 + hB, iB);
                WAIT_V(6);
                if (look && wid == 0) {
                    const unsigned long long mask = arrived(fx, fw);
                    if (lane == 0) { lavail[4] = (unsigned)mask; lavail[5] = (unsigned)(mask >> 32); }
                    WAIT_L(0);
                }
                CFENCE; BAR; CFENCE;
                unsigned lo = 0, hi = 0;
                if (look) { lo = lavail[4]; hi = lavail[5]; }
                MMA(1, 1, At, B1); BAR;
                if (look) {
                    const unsigned long long m = ((unsigned long long)(unsigned)__builtin_amdgcn_readfirstlane((int)hi) << 32) | (unsigned)__builtin_amdgcn_readfirstlane((int)lo);
                    ready |= m; seen |= m;
                }
                a1 = a3;
                if (last) break;
            }
            WAIT_V(0);
            if (wr == 0) BAR;
            BAR;
        }
        PT(own ? (is_copy ? 5 : 1) : 9);
    }

    const bool bad0 = (unsigned)__builtin_amdgcn_readfirstlane((int)ld_agent(tmo)) == call;
    const u32x4 nan4 = {0x7FC07FC0u, 0x7FC07FC0u, 0x7FC07FC0u, 0x7FC07FC0u};
    const unsigned pbase = PS_OFF + par * PS_BYTES + (unsigned)(mi * 8 + nj) * 131072u + (unsigned)wid * 16384u;
    if (!is_copy && rb != rank) {
#pragma unroll
        for (int ai = 0; ai < 2; ++ai)
#pragma unroll
            for (int bj = 0; bj < 2; ++bj)
#pragma unroll
                for (int m = 0; m < 4; ++m) {
                    const f32x4 v0 = acc[ai][bj][m][0], v1 = acc[ai][bj][m][1];
                    u32x4 w; w.x = pack_bf16(v0[0], v0[1]); w.y = pack_bf16(v0[2], v0[3]); w.z = pack_bf16(v1[0], v1[1]); w.w = pack_bf16(v1[2], v1[3]);
                    if (bad0) w = nan4;
                    store16<17>(w, rM, (unsigned)lane * 16u, pbase + (unsigned)((ai * 2 + bj) * 4 + m) * 1024u);
                }
        WAIT_V(0);
        if (lane == 0) st_sys((unsigned*)(a.ws[rb] + PF_OFF) + (unsigned)((rank * 2 + hh) * 8 + nj) * 32u + wid, call);
        PT(10); PEND();
        return;
    }
    const __amdgpu_buffer_rsrc_t rO = mk_rsrc(a.out, O_BYTES);
    const unsigned vo = (unsigned)(wr * 64 + fr) * WP + (unsigned)(wc * 32 + ((fq & 1) << 4) + ((fq >> 1) << 3)) * 2u;
    const unsigned so = (unsigned)hh * 256u * WP + (unsigned)(colrank * 2048 + nj * 256) * 2u;
    auto store_out = [&](int ai, int bj, int m, bool bad) {
        const f32x4 v0 = acc[ai][bj][m][0], v1 = acc[ai][bj][m][1];
        const unsigned ax = pack_bf16(v0[0], v0[1]), ay = pack_bf16(v0[2], v0[3]);
        const unsigned bx = pack_bf16(v1[0], v1[1]), by = pack_bf16(v1[2], v1[3]);
        const auto sx = __builtin_amdgcn_permlane16_swap(ax, bx, false, false);
        const auto sy = __builtin_amdgcn_permlane16_swap(ay, by, false, false);
        u32x4 w = {sx[0], sy[0], sx[1], sy[1]};
        if (bad) w = nan4;
        store16<OUT_AUX>(w, rO, vo, so + (unsigned)(ai * 128 + m * 16) * WP + (unsigned)bj * 256u);
    };
    if (is_copy) {
        const unsigned* const pf = (const unsigned*)(myws + PF_OFF) + (unsigned)((colrank * 2 + hh) * 8 + nj) * 32u + wid;
        const unsigned long long t0 = __builtin_amdgcn_s_memrealtime();
        bool bad = bad0;
        while ((int)((unsigned)__builtin_amdgcn_readfirstlane((int)ld_sys(pf)) - call) < 0) {
            if ((unsigned)__builtin_amdgcn_readfirstlane((int)ld_agent(tmo)) == call ||
                __builtin_amdgcn_s_memrealtime() - t0 > DEADLINE_TICKS) {
                if (lane == 0) st_agent(tmo, call);
                bad = true;
                break;
            }
            __builtin_amdgcn_s_sleep(2);
        }
        CFENCE;
        PT(6);
        const __amdgpu_buffer_rsrc_t rP = mk_rsrc(a.ws[colrank], WS_NEED);
        auto combine = [&](int u, const u32x4 w) {
            const int ai = u >> 3, bj = (u >> 2) & 1, m = u & 3;
            f32x4& v0 = acc[ai][bj][m][0]; f32x4& v1 = acc[ai][bj][m][1];
            v0[0] += __uint_as_float(w.x << 16); v0[1] += __uint_as_float(w.x & 0xFFFF0000u);
            v0[2] += __uint_as_float(w.y << 16); v0[3] += __uint_as_float(w.y & 0xFFFF0000u);
            v1[0] += __uint_as_float(w.z << 16); v1[1] += __uint_as_float(w.z & 0xFFFF0000u);
            v1[2] += __uint_as_float(w.w << 16); v1[3] += __uint_as_float(w.w & 0xFFFF0000u);
            store_out(ai, bj, m, bad);
        };
        if ((hh ? WINL : WINU) == 1) {
            u32x4 pv = __builtin_amdgcn_raw_buffer_load_b128(rP, (unsigned)lane * 16u, pbase, 17);
#pragma unroll
            for (int u = 0; u < 16; ++u) {
                WAIT_V(0);
                asm volatile("" : "+v"(pv));
                const u32x4 w = pv;
                if (u < 15) pv = __builtin_amdgcn_raw_buffer_load_b128(rP, (unsigned)lane * 16u, pbase + (unsigned)(u + 1) * 1024u, 17);
                SCHED;
                combine(u, w);
            }
        } else {
            u32x4 p0 = __builtin_amdgcn_raw_buffer_load_b128(rP, (unsigned)lane * 16u, pbase, 17);
            u32x4 p1 = __builtin_amdgcn_raw_buffer_load_b128(rP, (unsigned)lane * 16u, pbase + 1024u, 17);
#pragma unroll
            for (int u = 0; u < 16; u += 2) {
                WAIT_V(0);
                asm volatile("" : "+v"(p0), "+v"(p1));
                const u32x4 w0 = p0, w1 = p1;
                if (u < 14) {
                    p0 = __builtin_amdgcn_raw_buffer_load_b128(rP, (unsigned)lane * 16u, pbase + (unsigned)(u + 2) * 1024u, 17);
                    p1 = __builtin_amdgcn_raw_buffer_load_b128(rP, (unsigned)lane * 16u, pbase + (unsigned)(u + 3) * 1024u, 17);
                }
                SCHED;
                combine(u, w0); combine(u + 1, w1);
            }
        }
    } else {
#pragma unroll
        for (int ai = 0; ai < 2; ++ai)
#pragma unroll
            for (int m = 0; m < 4; ++m)
#pragma unroll
                for (int bj = 0; bj < 2; ++bj) store_out(ai, bj, m, bad0);
    }
}

}

extern "C" void kernel_launch(int rank, int nranks, ncclComm_t comm,
                              hipStream_t stream,
                              void* const* d_in, const int* in_sizes, int n_in,
                              void* d_out, int out_size,
                              const int* in_dtypes, int out_dtype,
                              void* const* d_out_all,
                              void* const* d_ws_all, size_t ws_bytes) {
    (void)comm; (void)in_dtypes; (void)out_dtype; (void)d_out_all;
    if (nranks != NR || n_in < 2 || rank < 0 || rank >= NR || ws_bytes < (size_t)WS_NEED ||
        in_sizes[0] != 4096 * 512 || in_sizes[1] != 512 * 16384 || out_size != 512 * 16384) {
        fprintf(stderr, "kernel_launch: unexpected problem shape\n");
        return;
    }
    static bool attr_set[64] = {};
    int dev = 0;
    HIP_CHECK(hipGetDevice(&dev));
    if (dev >= 0 && dev < 64 && !attr_set[dev]) {
        HIP_CHECK(hipFuncSetAttribute((const void*)fused_gemm_rs, hipFuncAttributeMaxDynamicSharedMemorySize, (int)LDS_BYTES));
        attr_set[dev] = true;
    }
    Args a;
    a.x = (const char*)d_in[0];
    a.w = (const char*)d_in[1];
    a.out = (char*)d_out;
    for (int p = 0; p < NR; ++p) a.ws[p] = (char*)d_ws_all[p];
    a.rank = rank;
    hipLaunchKernelGGL(fused_gemm_rs, dim3(NBLK), dim3(512), LDS_BYTES, stream, a);
    HIP_CHECK(hipGetLastError());
}
```

```cpp
#include <hip/hip_runtime.h>
#include <rccl/rccl.h>
#include <cstdio>
#include <cstdint>
#include <type_traits>

#define HIP_CHECK(expr) do { hipError_t _e = (expr); if (_e != hipSuccess) { \
    fprintf(stderr, "HIP error %s at %s:%d\n", hipGetErrorString(_e), __FILE__, __LINE__); \
    return; } } while (0)

namespace {

#define LAS __attribute__((address_space(3)))
typedef short bf16x8 __attribute__((ext_vector_type(8)));
typedef short s16x4 __attribute__((ext_vector_type(4)));
typedef short s16x8 __attribute__((ext_vector_type(8)));
typedef float f32x4 __attribute__((ext_vector_type(4)));
typedef float f32x2 __attribute__((ext_vector_type(2)));
typedef unsigned u32x4 __attribute__((ext_vector_type(4)));
typedef __bf16 bf16x2_t __attribute__((ext_vector_type(2)));

constexpr int NR = 8;
constexpr unsigned XP = 1024;
constexpr unsigned WP = 32768;
constexpr unsigned X_BYTES = 4096u * XP;
constexpr unsigned W_BYTES = 512u * WP;
constexpr unsigned O_BYTES = 512u * WP;
constexpr unsigned UNIT = 32768;
constexpr unsigned HTB = 16384;

constexpr unsigned CTR_OFF = 0;
constexpr unsigned TMO_OFF = 16384;
constexpr unsigned QC_OFF = 20480;
constexpr unsigned SF_OFF = 24576;
constexpr unsigned DF_OFF = 30720;
constexpr unsigned PF_OFF = 32768;
constexpr unsigned XF_OFF = 65536;
constexpr unsigned WF_OFF = 81920;
constexpr unsigned XD_OFF = 1u << 20;
constexpr unsigned WD_OFF = 33u << 20, WD_BYTES = 16u << 20;
constexpr unsigned SR_OFF = 65u << 20, SR_BYTES = 18u << 20;
constexpr unsigned PS_OFF = 101u << 20, PS_BYTES = 16u << 20;
constexpr unsigned DG_OFF = PS_OFF + 2 * PS_BYTES;
constexpr unsigned WS_NEED = DG_OFF + 56u * 262144u;

constexpr int NCOPY = 112, NGEMM = 128, NBLK = NCOPY + NGEMM;
#define OUT_AUX 17
#define WINU 2
#define WINL 1
#define THR 0
#define CWAIT 0
#define PLAY 1
#define ROT 1
constexpr int NUNITS = 176;
constexpr int NPUSHED = 8;
constexpr int NPULL = NUNITS - NPUSHED;
constexpr int NITEMS = 504;
constexpr unsigned LDS_BYTES = 131072 + 64;
constexpr unsigned long long DEADLINE_TICKS = 40000000ull;

struct Args {
    const char* x; const char* w; char* out;
    char* ws[NR];
    int rank;
};

__device__ __forceinline__ int lds_byte(int r, int c) {
    const int st = (r >> 4) * 2 + (c >> 5), rr = r & 15, cc = c & 31, ob = rr * 64 + cc * 2;
    return st * 1024 + (ob ^ (((ob >> 9) & 1) << 5));
}
__device__ __forceinline__ void stage_rc(int b, int& R, int& C) {
    const int st = b / 1024, sb = b % 1024, swz = sb ^ (((sb >> 9) & 1) << 5);
    R = (st >> 1) * 16 + swz / 64; C = (st & 1) * 32 + (swz % 64) / 2;
}
__device__ __forceinline__ unsigned pack_bf16(float lo, float hi) {
    f32x2 v = {lo, hi};
    return __builtin_bit_cast(unsigned, __builtin_convertvector(v, bf16x2_t));
}
__device__ __forceinline__ __amdgpu_buffer_rsrc_t mk_rsrc(const void* p, unsigned bytes) {
    return __builtin_amdgcn_make_buffer_rsrc((void*)p, 0, (int)bytes, 0x00020000);
}
template <int AUX>
__device__ __forceinline__ void store16(const u32x4 w, const __amdgpu_buffer_rsrc_t rs, unsigned voff, unsigned soff) {
    __builtin_amdgcn_raw_buffer_store_b128(w, rs, voff, soff, AUX);
    asm volatile("s_nop 1" :: "v"(w) : "memory");
}
__device__ __forceinline__ unsigned ld_sys(const unsigned* p) { return __hip_atomic_load(p, __ATOMIC_RELAXED, __HIP_MEMORY_SCOPE_SYSTEM); }
__device__ __forceinline__ unsigned ld_agent(const unsigned* p) { return __hip_atomic_load(p, __ATOMIC_RELAXED, __HIP_MEMORY_SCOPE_AGENT); }
__device__ __forceinline__ void st_sys(unsigned* p, unsigned v) { __hip_atomic_store(p, v, __ATOMIC_RELAXED, __HIP_MEMORY_SCOPE_SYSTEM); }
__device__ __forceinline__ void st_agent(unsigned* p, unsigned v) { __hip_atomic_store(p, v, __ATOMIC_RELAXED, __HIP_MEMORY_SCOPE_AGENT); }

#define PT(i) do {} while (0)
#define PEND() do {} while (0)
#define PBEG() do {} while (0)
#define WAIT_V(n) asm volatile("s_waitcnt vmcnt(" #n ")" ::: "memory")
#define WAIT_L(n) asm volatile("s_waitcnt lgkmcnt(" #n ")" ::: "memory")
#define BAR __builtin_amdgcn_s_barrier()
#define SCHED __builtin_amdgcn_sched_barrier(0)
#define CFENCE asm volatile("" ::: "memory")

#define SEQ0 0x1202091101100800ull
#define SEQ1 0x14040b1913030a18ull
#define SEQ2 0x16060d1b15050c1aull
#define SEQ3 0x1f1e0f1d0e17071cull
__device__ __forceinline__ int group_code(int g) {
    const unsigned long long q = g < 8 ? SEQ0 : g < 16 ? SEQ1 : g < 24 ? SEQ2 : SEQ3;
    return (int)(q >> ((g & 7) * 8)) & 31;
}
struct Cursor { int g, pos, item; };
__device__ __forceinline__ void decode_unit(Cursor& cu, int n, int dl, int& kind, int& step, int& idx) {
    int code = group_code(cu.g);
    while (n >= cu.pos + (code < 16 ? 4 : 7)) { cu.pos += code < 16 ? 4 : 7; ++cu.g; code = group_code(cu.g); }
    const int r = n - cu.pos;
    step = code & 7;
    if (code < 16) { kind = 0; idx = (code >> 3) * 4 + ((r + ROT * dl) & 3); }
    else { kind = (code >> 3) - 1; idx = r; }
}
__device__ __forceinline__ void decode_item(Cursor& cu, int t, int& n, int& dl) {
    int code = group_code(cu.g);
    for (;;) {
        const int cnt = cu.g < 2 ? 0 : code < 16 ? 28 : 7;
        if (t < cu.item + cnt) break;
        cu.item += cnt; cu.pos += code < 16 ? 4 : 7; ++cu.g; code = group_code(cu.g);
    }
    const int r = t - cu.item;
    if (code < 16) { const int q = r / 7; n = cu.pos + q; dl = 1 + r - q * 7; }
    else { n = cu.pos + r; dl = 0; }
}
__device__ __forceinline__ unsigned src_slot(int kind, int step, int idx, int dl) {
    return kind == 0 ? (unsigned)((dl - 1) * 64 + step * 8 + idx) : (unsigned)(448 + (kind - 1) * 56 + step * 7 + idx);
}

__global__ __launch_bounds__(512, 2) void fused_gemm_rs(Args a) {
    extern __shared__ __attribute__((aligned(16))) unsigned char shm[];
    LAS unsigned char* lds = (LAS unsigned char*)shm;
    const int tid = threadIdx.x, wid = __builtin_amdgcn_readfirstlane(tid >> 6), lane = tid & 63;
    const int b = blockIdx.x, rank = a.rank & 7;
    char* const myws = a.ws[rank];
    unsigned* const tmo = (unsigned*)(myws + TMO_OFF);

    unsigned voffA, voffB;
    { int R, C; stage_rc(tid * 16, R, C); voffA = (unsigned)R * XP + (unsigned)C * 2u;
      const int row = tid >> 4, chp = tid & 15, f = ((row & 3) << 2) | ((row >> 2) & 3);
      voffB = (unsigned)row * WP + (unsigned)(chp ^ f) * 16u; }
    const unsigned voffS = (unsigned)tid * 16u;

    const bool is_copy = b < NCOPY;
    const int dd = 1 + b % 7, c = b / 7;
    const int peer = (rank + dd) & 7;
    u32x4 r[4], rn[4];
    auto load_unit = [&](int kind, int step, int idx, int dest, u32x4 (&v)[4]) {
        if (kind == 0) {
            const __amdgpu_buffer_rsrc_t rs = mk_rsrc(a.w, W_BYTES);
            const unsigned so = (unsigned)step * 64u * WP + (unsigned)(dest * 2048 + idx * 256) * 2u;
#pragma unroll
            for (int h = 0; h < 2; ++h)
#pragma unroll
                for (int i = 0; i < 2; ++i)
                    v[h * 2 + i] = __builtin_amdgcn_raw_buffer_load_b128(rs, voffB, so + h * 256u + i * 32u * WP, 0);
        } else {
            const int tm = ((rank + 1 + idx) & 7) * 2 + (kind - 1);
            const __amdgpu_buffer_rsrc_t rs = mk_rsrc(a.x, X_BYTES);
            const unsigned so = (unsigned)tm * 256u * XP + (unsigned)step * 128u;
#pragma unroll
            for (int h = 0; h < 2; ++h)
#pragma unroll
                for (int i = 0; i < 2; ++i)
                    v[h * 2 + i] = __builtin_amdgcn_raw_buffer_load_b128(rs, voffA, so + h * 128u * XP + i * 64u * XP, 0);
        }
    };
    int in = 0, idl = 0, pu = 0;
    Cursor cu = {0, 0, 0};
    if (is_copy) {
        int kind, step, idx;
        if (c < 8) { decode_item(cu, b, in, idl); decode_unit(cu, in, idl, kind, step, idx); load_unit(kind, step, idx, (rank + idl) & 7, r); }
        else { decode_unit(cu, c - 8, dd, kind, step, idx); load_unit(0, step, idx, peer, r); pu = step * 8 + idx; }
    }

    unsigned* const ctr = (unsigned*)(myws + CTR_OFF + (unsigned)b * 64u);
    const unsigned call = (unsigned)__builtin_amdgcn_readfirstlane((int)ld_agent(ctr)) + 1u;
    WAIT_V(0);
    __syncthreads();
    if (tid == 0) st_agent(ctr, call);
    const unsigned par = call & 1u;
    PT(0); PBEG();
    const __amdgpu_buffer_rsrc_t rM = mk_rsrc(myws, WS_NEED);

    int mi, nj, colrank;
    if (is_copy) {
        colrank = peer; mi = rank * 2 + (c >> 3); nj = c & 7;
        if (c < 8) {
#pragma unroll 1
            for (int k = 0;; ++k) {
                int kind, step, idx; decode_unit(cu, in, idl, kind, step, idx);
                const unsigned doff = SR_OFF + par * SR_BYTES + src_slot(kind, step, idx, idl) * UNIT;
#pragma unroll
                for (int u = 0; u < 4; ++u)
                    store16<17>(r[u], rM, voffS, doff + (unsigned)u * 8192u);
                const int fn = in, fdl = idl;
                const bool more = k + 1 < NITEMS / 56;
                if (more) { decode_item(cu, b + (k + 1) * 56, in, idl); int k2, s2, i2; decode_unit(cu, in, idl, k2, s2, i2); load_unit(k2, s2, i2, (rank + idl) & 7, rn); }
                WAIT_V(0);
                asm volatile("" : "+v"(rn[0]), "+v"(rn[1]), "+v"(rn[2]), "+v"(rn[3]));
                __syncthreads();
                if (tid == 0) {
                    const unsigned fo = SF_OFF + (unsigned)(rank * 192 + fn) * 4u;
                    if (fdl) st_sys((unsigned*)(a.ws[(rank + fdl) & 7] + fo), call);
                    else {
#pragma unroll
                        for (int p = 1; p < 8; ++p) st_sys((unsigned*)(a.ws[(rank + p) & 7] + fo), call);
                    }
                }
                if (k == 0) PT(1);
                if (!more) break;
#pragma unroll
                for (int u = 0; u < 4; ++u) r[u] = rn[u];
            }
        } else {
            const int cp = c - 8, dl = 8 - dd;
            const __amdgpu_buffer_rsrc_t rP = mk_rsrc(a.ws[peer], WS_NEED);
            unsigned qoff = QC_OFF + (unsigned)(dd * 8 + wid) * 64u;
            asm volatile("" : "+v"(qoff));
            unsigned* const qc = (unsigned*)(myws + qoff);
            const unsigned qbase = (call - 1u) * (unsigned)NPULL - 8u;
            const unsigned* const sf = (const unsigned*)(myws + SF_OFF) + peer * 192 + lane;
            int su = NPUSHED;
            unsigned f0, f1, f2;
            auto sf_issue = [&]() { f0 = ld_sys(sf); f1 = ld_sys(sf + 64); f2 = ld_sys(sf + 128); };
            auto sf_eval = [&]() {
                const unsigned long long m0 = ~__builtin_amdgcn_ballot_w64((int)(f0 - call) >= 0 || lane < NPUSHED);
                const unsigned long long m1 = ~__builtin_amdgcn_ballot_w64((int)(f1 - call) >= 0);
                const unsigned long long m2 = ~__builtin_amdgcn_ballot_w64((int)(f2 - call) >= 0 || lane >= NUNITS - 128);
                su = m0 ? __builtin_ctzll(m0) : m1 ? 64 + __builtin_ctzll(m1) : m2 ? 128 + __builtin_ctzll(m2) : 192;
            };
            auto sf_wait = [&](int n) {
                if (n < su) return;
                const unsigned long long t0 = __builtin_amdgcn_s_memrealtime();
                for (;;) {
                    sf_issue(); sf_eval();
                    if (n < su) break;
                    if ((unsigned)__builtin_amdgcn_readfirstlane((int)ld_agent(tmo)) == call ||
                        __builtin_amdgcn_s_memrealtime() - t0 > DEADLINE_TICKS) {
                        if (lane == 0) st_agent(tmo, call);
                        su = 192; break;
                    }
                    __builtin_amdgcn_s_sleep(2);
                }
            };
            auto unit = [&](int q, int& n, unsigned& so, unsigned& dsto, unsigned& flo) {
                n = NPUSHED + q;
                int kind, step, idx; decode_unit(cu, n, dl, kind, step, idx);
                so = SR_OFF + par * SR_BYTES + src_slot(kind, step, idx, dl) * UNIT;
                if (kind == 0) {
                    dsto = WD_OFF + par * WD_BYTES + (unsigned)((peer * 8 + step) * 8 + idx) * UNIT;
                    flo = WF_OFF + (unsigned)(idx * 64 + peer * 8 + step) * 16u + (unsigned)wid * 2u;
                } else {
                    const int tm = ((peer + 1 + idx) & 7) * 2 + (kind - 1);
                    dsto = XD_OFF + (unsigned)((peer * 8 + step) * 16 + tm) * UNIT;
                    flo = XF_OFF + (unsigned)(tm * 64 + peer * 8 + step) * 16u + (unsigned)wid * 2u;
                }
            };
            auto set_flag = [&](const __amdgpu_buffer_rsrc_t rs, unsigned flo) {
                if (lane == 0) __builtin_amdgcn_raw_buffer_store_b16((unsigned short)call, rs, 0, flo, 17);
            };
            const unsigned poff = WD_OFF + par * WD_BYTES + (unsigned)(rank * 64 + pu) * UNIT;
            store16<17>(r[0], rP, voffS, poff); WAIT_V(0);
            store16<17>(r[1], rP, voffS, poff + 8192u); WAIT_V(0);
            store16<17>(r[2], rP, voffS, poff + 16384u); sf_issue(); WAIT_V(0);
            store16<17>(r[3], rP, voffS, poff + 24576u);
            sf_eval();
            int n0, n1 = 0; unsigned so0, do0, fl0, so1 = 0, do1 = 0, fl1 = 0, flp = 0;
            unit(cp, n0, so0, do0, fl0);
            sf_wait(n0);
            u32x4 v = __builtin_amdgcn_raw_buffer_load_b128(rP, voffS, so0, 17);
            bool pend = false;
            unsigned qv = 0;
#pragma unroll 1
            for (int k = 0;; ++k) {
                bool more = false;
#pragma unroll
                for (int u = 0; u < 4; ++u) {
                    WAIT_V(0);
                    asm volatile("" : "+v"(v));
                    const u32x4 w = v;
                    if (u < 3) v = __builtin_amdgcn_raw_buffer_load_b128(rP, voffS, so0 + (unsigned)(u + 1) * 8192u, 17);
                    else if (more) {
                        if (pend) { sf_eval(); pend = false; }
                        sf_wait(n1);
                        v = __builtin_amdgcn_raw_buffer_load_b128(rP, voffS, so1, 17);
                        if (su < NUNITS) { sf_issue(); pend = true; }
                    }
                    store16<17>(w, rM, voffS, do0 + (unsigned)u * 8192u);
                    if (u == 0) {
                        if (k == 0) { set_flag(rP, WF_OFF + (unsigned)((pu & 7) * 64 + rank * 8 + (pu >> 3)) * 16u + (unsigned)wid * 2u); PT(1); }
                        else set_flag(rM, flp);
                        if (lane == 0) qv = __hip_atomic_fetch_add(qc, 1u, __ATOMIC_RELAXED, __HIP_MEMORY_SCOPE_AGENT);
                    }
                    if (u == 2) {
                        const int qn = __builtin_amdgcn_readfirstlane((int)(qv - qbase));
                        more = qn < NPULL;
                        if (more) unit(qn, n1, so1, do1, fl1);
                    }
                }
                flp = fl0;
                if (!more) break;
                so0 = so1; do0 = do1; fl0 = fl1;
                if (k == 4) PT(2);
                if (k == 10) PT(3);
            }
            WAIT_V(0);
            set_flag(rM, flp);
        }
        PT(4);
    } else {
        const int g = b - NCOPY, xcd = g & 7, idx = g >> 3;
        mi = 4 * (xcd >> 1) + (idx & 3); nj = 4 * (xcd & 1) + (idx >> 2); colrank = rank;
    }
    const int rb = mi >> 1, hh = mi & 1;

    const int wr = wid >> 2, wc = wid & 3, fr = lane & 15, fq = lane >> 4;
    const unsigned ldsw = (unsigned)wid * 1024u;
    const int aoff = lds_byte(wr * 64 + fr, fq * 8);
    unsigned boff[2][2];
    { const int q = fr >> 2, p = lane & 3;
#pragma unroll
      for (int n = 0; n < 2; ++n)
#pragma unroll
          for (int t = 0; t < 2; ++t) {
              const int ch = 4 * wc + 2 * n + (p >> 1), f = (q << 2) | ((2 * fq + t) & 3);
              boff[n][t] = (unsigned)(size_t)lds + 65536u + 256u * (unsigned)(8 * fq + 4 * t + q) + 16u * (unsigned)(ch ^ f) + 8u * (unsigned)(p & 1);
          } }
#define SA(bb, h) (((bb) * 2 + (h)) * HTB)
#define SB(bb, h) ((4 + (bb) * 2 + (h)) * HTB)
#define SBR(bb, h) (((bb) * 2 + (h)) * HTB)
#define STAGE(bufoff, rs, voff, soff, istr) do { _Pragma("unroll") for (int _i = 0; _i < 2; ++_i) \
        __builtin_amdgcn_raw_ptr_buffer_load_lds(rs, (LAS void*)(lds + (bufoff) + ldsw + _i * 8192), 16, voff, (soff) + _i * (istr), 0, 16); } while (0)
#define LDA(dst, bb, h) do { _Pragma("unroll") for (int m = 0; m < 4; ++m) _Pragma("unroll") for (int k = 0; k < 2; ++k) \
        dst[m][k] = *(const LAS bf16x8*)(lds + SA(bb, h) + aoff + m * 2048 + k * 1024); } while (0)
#define TRR(d, addr, off) asm volatile("ds_read_b64_tr_b16 %0, %1 offset:%2" : "=&v"(d) : "v"(addr), "i"(off) : "memory")
#define LDB(dst, bb, h) do { _Pragma("unroll") for (int n = 0; n < 2; ++n) _Pragma("unroll") for (int k = 0; k < 2; ++k) { \
        TRR(dst[n][k][0], boff[n][0], SBR(bb, h) + k * 8192); TRR(dst[n][k][1], boff[n][1], SBR(bb, h) + k * 8192); } } while (0)
#define PKB(v) __builtin_shufflevector((v)[0], (v)[1], 0, 1, 2, 3, 4, 5, 6, 7)
#define MMA(ai, bj, At, Bt) do { SCHED; __builtin_amdgcn_s_setprio(1); _Pragma("unroll") for (int m = 0; m < 4; ++m) _Pragma("unroll") for (int n = 0; n < 2; ++n) \
        _Pragma("unroll") for (int k = 0; k < 2; ++k) acc[ai][bj][m][n] = __builtin_amdgcn_mfma_f32_16x16x32_bf16(PKB(Bt[n][k]), At[m][k], acc[ai][bj][m][n], 0, 0, 0); \
        __builtin_amdgcn_s_setprio(0); } while (0)

    f32x4 acc[2][2][4][2];
#pragma unroll
    for (int i0 = 0; i0 < 2; ++i0)
#pragma unroll
        for (int i1 = 0; i1 < 2; ++i1)
#pragma unroll
            for (int m = 0; m < 4; ++m)
#pragma unroll
                for (int n = 0; n < 2; ++n) acc[i0][i1][m][n] = (f32x4){0.f, 0.f, 0.f, 0.f};
    bf16x8 At[4][2]; s16x4 B0[2][2][2], B1[2][2][2];

    const int np = (rb == rank) ? 7 : 6;
    const unsigned long long valid = ~((0xFFull << (rank * 8)) | (0xFFull << (rb * 8)));
    const int nph = !is_copy || c < 8 ? 2 : 0;
    const unsigned dgo = DG_OFF + (unsigned)(b % 56) * 262144u + (unsigned)wid * 32768u;
    unsigned* const dgf = (unsigned*)(myws + DF_OFF) + (b % 56) * 8 + wid;
#pragma unroll 1
    for (int ph = 0; ph < nph; ++ph) {
        const bool own = is_copy || ph == 0;
        const int mic = is_copy ? rank * 2 + 1 - ph : mi;
        const int nt = own ? 8 : np * 8;
        const __amdgpu_buffer_rsrc_t rA = mk_rsrc(own ? a.x : (const char*)myws, own ? X_BYTES : WS_NEED);
        const __amdgpu_buffer_rsrc_t rB = mk_rsrc(own ? a.w : (const char*)myws, own ? W_BYTES : WS_NEED);
        const unsigned vA = own ? voffA : voffS, vB = own ? voffB : voffS;
        const unsigned iA = own ? 64u * XP : 8192u, iB = own ? 32u * WP : 8192u;
        const unsigned hA = own ? 128u * XP : HTB, hB = own ? 256u : HTB;
        auto addrA = [&](int id) -> unsigned {
            return own ? (unsigned)mic * 256u * XP + (unsigned)id * 128u
                       : XD_OFF + (unsigned)(id * 16 + mi) * UNIT;
        };
        auto addrB = [&](int id) -> unsigned {
            return own ? (unsigned)id * 64u * WP + (unsigned)(colrank * 2048 + nj * 256) * 2u
                       : WD_OFF + par * WD_BYTES + (unsigned)(id * 8 + nj) * UNIT;
        };
        unsigned long long ready = own ? 0xFFull : 0ull, seen = ready;
        const unsigned xf = XF_OFF + (unsigned)(mi * 64) * 16u, wf = WF_OFF + (unsigned)(nj * 64) * 16u;
        LAS volatile unsigned* const lavail = (LAS volatile unsigned*)(lds + 131072);
        int pslot = 0;
        const short c16 = (short)call;
        const s16x8 c8 = {c16, c16, c16, c16, c16, c16, c16, c16};
        auto arrived = [&](const u32x4 fx, const u32x4 fw) -> unsigned long long {
            const s16x8 dx = __builtin_bit_cast(s16x8, fx) - c8, dw = __builtin_bit_cast(s16x8, fw) - c8;
            const u32x4 dd4 = __builtin_bit_cast(u32x4, dx) | __builtin_bit_cast(u32x4, dw);
            const bool ok = ((dd4.x | dd4.y | dd4.z | dd4.w) & 0x80008000u) == 0u;
            return __builtin_amdgcn_ballot_w64(ok) & valid & ~seen;
        };
        auto pop = [&]() -> int { const int id = __builtin_ctzll(ready); ready &= ready - 1ull; return id; };
        int remaining = nt;
#pragma unroll 1
        while (remaining > 0) {
#pragma unroll 1
            while (__builtin_popcountll(ready) < 2) {
                if (wid == 0) {
                    const unsigned long long t0 = __builtin_amdgcn_s_memrealtime();
                    unsigned long long mask;
                    for (;;) {
                        CFENCE;
                        const u32x4 fx = __builtin_amdgcn_raw_buffer_load_b128(rM, (unsigned)lane * 16u, xf, 17);
                        const u32x4 fw = __builtin_amdgcn_raw_buffer_load_b128(rM, (unsigned)lane * 16u, wf, 17);
                        mask = arrived(fx, fw);
                        if (mask != 0ull) break;
                        if ((unsigned)__builtin_amdgcn_readfirstlane((int)ld_agent(tmo)) == call ||
                            __builtin_amdgcn_s_memrealtime() - t0 > DEADLINE_TICKS) {
                            if (lane == 0) st_agent(tmo, call);
                            mask = valid & ~seen; break;
                        }
                        __builtin_amdgcn_s_sleep(2);
                    }
                    if (lane == 0) { lavail[pslot] = (unsigned)mask; lavail[pslot + 1] = (unsigned)(mask >> 32); }
                    WAIT_V(0); WAIT_L(0);
                }
                CFENCE; BAR; CFENCE;
                const unsigned lo = (unsigned)__builtin_amdgcn_readfirstlane((int)lavail[pslot]), hi = (unsigned)__builtin_amdgcn_readfirstlane((int)lavail[pslot + 1]);
                pslot ^= 2;
                const unsigned long long m = ((unsigned long long)hi << 32) | lo;
                ready |= m; seen |= m;
            }
            const int id0 = pop(), id1 = pop();
            { const unsigned a0 = addrA(id0), b0 = addrB(id0), a1 = addrA(id1), b1 = addrB(id1);
              STAGE(SB(0, 0), rB, vB, b0, iB); STAGE(SA(0, 0), rA, vA, a0, iA); STAGE(SB(0, 1), rB, vB, b0 + hB, iB); STAGE(SA(0, 1), rA, vA, a0 + hA, iA);
              if (wr == 1) BAR;
              WAIT_V(4); BAR;
              STAGE(SB(1, 0), rB, vB, b1, iB); STAGE(SA(1, 0), rA, vA, a1, iA); STAGE(SB(1, 1), rB, vB, b1 + hB, iB);
              WAIT_V(6); BAR; }
            unsigned a1 = addrA(id1);
#pragma unroll 1
            for (;;) {
                remaining -= 2;
                const bool last = __builtin_popcountll(ready) < 2;
                const int t2 = last ? id0 : pop();
                const int t3 = last ? id0 : pop();
                const bool look = !own && (valid & ~seen) != 0ull;
                const unsigned a2 = addrA(t2), b2 = addrB(t2);
                const unsigned a3 = addrA(t3), b3 = addrB(t3);
                u32x4 fx, fw;
                LDB(B0, 0, 0); SCHED; LDA(At, 0, 0); STAGE(SA(1, 1), rA, vA, a1 + hA, iA);
                WAIT_L(8); BAR; WAIT_L(0); MMA(0, 0, At, B0); BAR; SCHED;
                LDB(B1, 0, 1); STAGE(SB(0, 0), rB, vB, b2, iB);
                BAR; WAIT_L(0); MMA(0, 1, At, B1); BAR;
                LDA(At, 0, 1); STAGE(SA(0, 0), rA, vA, a2, iA);
                BAR; WAIT_L(0); MMA(1, 0, At, B0); BAR; SCHED;
                STAGE(SB(0, 1), rB, vB, b2 + hB, iB);
                WAIT_V(6); BAR; MMA(1, 1, At, B1); BAR;
                if (look && wid == 0) {
                    fx = __builtin_amdgcn_raw_buffer_load_b128(rM, (unsigned)lane * 16u, xf, 17);
                    fw = __builtin_amdgcn_raw_buffer_load_b128(rM, (unsigned)lane * 16u, wf, 17);
                }
                LDB(B0, 1, 0); SCHED; LDA(At, 1, 0); STAGE(SA(0, 1), rA, vA, a2 + hA, iA);
                WAIT_L(8); BAR; WAIT_L(0); MMA(0, 0, At, B0); BAR; SCHED;
                LDB(B1, 1, 1); STAGE(SB(1, 0), rB, vB, b3, iB);
                BAR; WAIT_L(0); MMA(0, 1, At, B1); BAR;
                LDA(At, 1, 1); STAGE(SA(1, 0), rA, vA, a3, iA);
                BAR; WAIT_L(0); MMA(1, 0, At, B0); BAR; SCHED;
                STAGE(SB(1, 1), rB, vB, b3 + hB, iB);
                WAIT_V(6);
                if (look && wid == 0) {
                    const unsigned long long mask = arrived(fx, fw);
                    if (lane == 0) { lavail[4] = (unsigned)mask; lavail[5] = (unsigned)(mask >> 32); }
                    WAIT_L(0);
                }
                CFENCE; BAR; CFENCE;
                unsigned lo = 0, hi = 0;
                if (look) { lo = lavail[4]; hi = lavail[5]; }
                MMA(1, 1, At, B1); BAR;
                if (look) {
                    const unsigned long long m = ((unsigned long long)(unsigned)__builtin_amdgcn_readfirstlane((int)hi) << 32) | (unsigned)__builtin_amdgcn_readfirstlane((int)lo);
                    ready |= m; seen |= m;
                }
                a1 = a3;
                if (last) break;
            }
            WAIT_V(0);
            if (wr == 0) BAR;
            BAR;
        }
        PT(own ? (is_copy ? 5 : 1) : 9);
        if (is_copy && ph == 0) {
#pragma unroll
            for (int u = 0; u < 32; ++u) {
                f32x4& v = acc[u >> 4][(u >> 3) & 1][(u >> 1) & 3][u & 1];
                store16<17>(__builtin_bit_cast(u32x4, v), rM, (unsigned)lane * 16u, dgo + (unsigned)u * 1024u);
                v = (f32x4){0.f, 0.f, 0.f, 0.f};
            }
            WAIT_V(0);
            if (lane == 0) st_agent(dgf, call);
        }
    }
    if (nph == 0) {
        const unsigned long long t0 = __builtin_amdgcn_s_memrealtime();
        while ((unsigned)__builtin_amdgcn_readfirstlane((int)ld_agent(dgf)) != call) {
            if ((unsigned)__builtin_amdgcn_readfirstlane((int)ld_agent(tmo)) == call ||
                __builtin_amdgcn_s_memrealtime() - t0 > DEADLINE_TICKS) {
                if (lane == 0) st_agent(tmo, call);
                break;
            }
            __builtin_amdgcn_s_sleep(2);
        }
        CFENCE;
#pragma unroll
        for (int u = 0; u < 32; ++u)
            acc[u >> 4][(u >> 3) & 1][(u >> 1) & 3][u & 1] =
                __builtin_bit_cast(f32x4, __builtin_amdgcn_raw_buffer_load_b128(rM, (unsigned)lane * 16u, dgo + (unsigned)u * 1024u, 17));
        PT(5);
    }

    const bool bad0 = (unsigned)__builtin_amdgcn_readfirstlane((int)ld_agent(tmo)) == call;
    const u32x4 nan4 = {0x7FC07FC0u, 0x7FC07FC0u, 0x7FC07FC0u, 0x7FC07FC0u};
    constexpr unsigned PSTR = 8192u;
    const unsigned pbase = PS_OFF + par * PS_BYTES + (unsigned)(mi * 8 + nj) * 131072u + (unsigned)wid * 1024u;
    if (!is_copy && rb != rank) {
#pragma unroll
        for (int ai = 0; ai < 2; ++ai)
#pragma unroll
            for (int bj = 0; bj < 2; ++bj)
#pragma unroll
                for (int m = 0; m < 4; ++m) {
                    const f32x4 v0 = acc[ai][bj][m][0], v1 = acc[ai][bj][m][1];
                    u32x4 w; w.x = pack_bf16(v0[0], v0[1]); w.y = pack_bf16(v0[2], v0[3]); w.z = pack_bf16(v1[0], v1[1]); w.w = pack_bf16(v1[2], v1[3]);
                    if (bad0) w = nan4;
                    store16<17>(w, rM, (unsigned)lane * 16u, pbase + (unsigned)((ai * 2 + bj) * 4 + m) * PSTR);
                }
        WAIT_V(0);
        if (lane == 0) st_sys((unsigned*)(a.ws[rb] + PF_OFF) + (unsigned)((rank * 2 + hh) * 8 + nj) * 32u + wid, call);
        PT(10); PEND();
        return;
    }
    const __amdgpu_buffer_rsrc_t rO = mk_rsrc(a.out, O_BYTES);
    const unsigned vo = (unsigned)(wr * 64 + fr) * WP + (unsigned)(wc * 32 + ((fq & 1) << 4) + ((fq >> 1) << 3)) * 2u;
    const unsigned so = (unsigned)hh * 256u * WP + (unsigned)(colrank * 2048 + nj * 256) * 2u;
    auto store_out = [&](int ai, int bj, int m, bool bad) {
        const f32x4 v0 = acc[ai][bj][m][0], v1 = acc[ai][bj][m][1];
        const unsigned ax = pack_bf16(v0[0], v0[1]), ay = pack_bf16(v0[2], v0[3]);
        const unsigned bx = pack_bf16(v1[0], v1[1]), by = pack_bf16(v1[2], v1[3]);
        const auto sx = __builtin_amdgcn_permlane16_swap(ax, bx, false, false);
        const auto sy = __builtin_amdgcn_permlane16_swap(ay, by, false, false);
        u32x4 w = {sx[0], sy[0], sx[1], sy[1]};
        if (bad) w = nan4;
        store16<OUT_AUX>(w, rO, vo, so + (unsigned)(ai * 128 + m * 16) * WP + (unsigned)bj * 256u);
    };
    if (is_copy) {
        const unsigned* const pf = (const unsigned*)(myws + PF_OFF) + (unsigned)((colrank * 2 + hh) * 8 + nj) * 32u + wid;
        const unsigned long long t0 = __builtin_amdgcn_s_memrealtime();
        bool bad = bad0;
        while ((int)((unsigned)__builtin_amdgcn_readfirstlane((int)ld_sys(pf)) - call) < 0) {
            if ((unsigned)__builtin_amdgcn_readfirstlane((int)ld_agent(tmo)) == call ||
                __builtin_amdgcn_s_memrealtime() - t0 > DEADLINE_TICKS) {
                if (lane == 0) st_agent(tmo, call);
                bad = true;
                break;
            }
            __builtin_amdgcn_s_sleep(2);
        }
        CFENCE;
        PT(6);
        const __amdgpu_buffer_rsrc_t rP = mk_rsrc(a.ws[colrank], WS_NEED);
        auto combine = [&](int u, const u32x4 w) {
            const int ai = u >> 3, bj = (u >> 2) & 1, m = u & 3;
            f32x4& v0 = acc[ai][bj][m][0]; f32x4& v1 = acc[ai][bj][m][1];
            v0[0] += __uint_as_float(w.x << 16); v0[1] += __uint_as_float(w.x & 0xFFFF0000u);
            v0[2] += __uint_as_float(w.y << 16); v0[3] += __uint_as_float(w.y & 0xFFFF0000u);
            v1[0] += __uint_as_float(w.z << 16); v1[1] += __uint_as_float(w.z & 0xFFFF0000u);
            v1[2] += __uint_as_float(w.w << 16); v1[3] += __uint_as_float(w.w & 0xFFFF0000u);
            store_out(ai, bj, m, bad);
        };
        auto pull = [&](auto winc) {
            constexpr int W = decltype(winc)::value;
            u32x4 pv[W];
#pragma unroll
            for (int j = 0; j < W; ++j) pv[j] = __builtin_amdgcn_raw_buffer_load_b128(rP, (unsigned)lane * 16u, pbase + (unsigned)j * PSTR, 17);
#pragma unroll
            for (int u = 0; u < 16; ++u) {
                const int nl = (u + W - 1 < 16 ? W - 1 : 15 - u), ns = u < W ? u : W;
                switch (nl + ns) {
                    case 0: WAIT_V(0); break; case 1: WAIT_V(1); break; case 2: WAIT_V(2); break; case 3: WAIT_V(3); break;
                    case 4: WAIT_V(4); break; case 5: WAIT_V(5); break; case 6: WAIT_V(6); break; default: WAIT_V(7); break;
                }
                asm volatile("" : "+v"(pv[u % W]));
                const u32x4 w = pv[u % W];
                if (u + W < 16) pv[u % W] = __builtin_amdgcn_raw_buffer_load_b128(rP, (unsigned)lane * 16u, pbase + (unsigned)(u + W) * PSTR, 17);
                SCHED;
                combine(u, w);
                SCHED;
            }
        };
        if (hh == 0) pull(std::integral_constant<int, WINU>{}); else pull(std::integral_constant<int, WINL>{});
    } else {
#pragma unroll
        for (int ai = 0; ai < 2; ++ai)
#pragma unroll
            for (int m = 0; m < 4; ++m)
#pragma unroll
                for (int bj = 0; bj < 2; ++bj) store_out(ai, bj, m, bad0);
    }
}

}

extern "C" void kernel_launch(int rank, int nranks, ncclComm_t comm,
                              hipStream_t stream,
                              void* const* d_in, const int* in_sizes, int n_in,
                              void* d_out, int out_size,
                              const int* in_dtypes, int out_dtype,
                              void* const* d_out_all,
                              void* const* d_ws_all, size_t ws_bytes) {
    (void)comm; (void)in_dtypes; (void)out_dtype; (void)d_out_all;
    if (nranks != NR || n_in < 2 || rank < 0 || rank >= NR || ws_bytes < (size_t)WS_NEED ||
        in_sizes[0] != 4096 * 512 || in_sizes[1] != 512 * 16384 || out_size != 512 * 16384) {
        fprintf(stderr, "kernel_launch: unexpected problem shape\n");
        return;
    }
    static bool attr_set[64] = {};
    int dev = 0;
    HIP_CHECK(hipGetDevice(&dev));
    if (dev >= 0 && dev < 64 && !attr_set[dev]) {
        HIP_CHECK(hipFuncSetAttribute((const void*)fused_gemm_rs, hipFuncAttributeMaxDynamicSharedMemorySize, (int)LDS_BYTES));
        attr_set[dev] = true;
    }
    Args a;
    a.x = (const char*)d_in[0];
    a.w = (const char*)d_in[1];
    a.out = (char*)d_out;
    for (int p = 0; p < NR; ++p) a.ws[p] = (char*)d_ws_all[p];
    a.rank = rank;
    hipLaunchKernelGGL(fused_gemm_rs, dim3(NBLK), dim3(512), LDS_BYTES, stream, a);
    HIP_CHECK(hipGetLastError());
}
```

```cpp
#include <hip/hip_runtime.h>
#include <rccl/rccl.h>
#include <cstdio>
#include <cstdint>
#include <type_traits>

#define HIP_CHECK(expr) do { hipError_t _e = (expr); if (_e != hipSuccess) { \
    fprintf(stderr, "HIP error %s at %s:%d\n", hipGetErrorString(_e), __FILE__, __LINE__); \
    return; } } while (0)

namespace {

#define LAS __attribute__((address_space(3)))
typedef short bf16x8 __attribute__((ext_vector_type(8)));
typedef short s16x4 __attribute__((ext_vector_type(4)));
typedef short s16x8 __attribute__((ext_vector_type(8)));
typedef float f32x4 __attribute__((ext_vector_type(4)));
typedef float f32x2 __attribute__((ext_vector_type(2)));
typedef unsigned u32x4 __attribute__((ext_vector_type(4)));
typedef __bf16 bf16x2_t __attribute__((ext_vector_type(2)));

constexpr int NR = 8;
constexpr unsigned XP = 1024;
constexpr unsigned WP = 32768;
constexpr unsigned X_BYTES = 4096u * XP;
constexpr unsigned W_BYTES = 512u * WP;
constexpr unsigned O_BYTES = 512u * WP;
constexpr unsigned UNIT = 32768;
constexpr unsigned HTB = 16384;

constexpr unsigned CTR_OFF = 0;
constexpr unsigned TMO_OFF = 16384;
constexpr unsigned QC_OFF = 20480;
constexpr unsigned SF_OFF = 24576;
constexpr unsigned DF_OFF = 30720;
constexpr unsigned PF_OFF = 32768;
constexpr unsigned XF_OFF = 65536;
constexpr unsigned WF_OFF = 81920;
constexpr unsigned XD_OFF = 1u << 20;
constexpr unsigned WD_OFF = 33u << 20, WD_BYTES = 16u << 20;
constexpr unsigned SR_OFF = 65u << 20, SR_BYTES = 18u << 20;
constexpr unsigned PS_OFF = 101u << 20, PS_BYTES = 16u << 20;
constexpr unsigned DG_OFF = PS_OFF + 2 * PS_BYTES;
constexpr unsigned WS_NEED = DG_OFF + 56u * 262144u;

constexpr int NCOPY = 112, NGEMM = 128, NBLK = NCOPY + NGEMM;
constexpr int OUT_AUX = 17;
constexpr int WINU = 2, WINL = 1;
constexpr int ROT = 1;
constexpr int NUNITS = 176;
constexpr int NPUSHED = 8;
constexpr int NPULL = NUNITS - NPUSHED;
constexpr int NITEMS = 504;
constexpr unsigned LDS_BYTES = 131072 + 64;
constexpr unsigned long long DEADLINE_TICKS = 40000000ull;

struct Args {
    const char* x; const char* w; char* out;
    char* ws[NR];
    int rank;
};

__device__ __forceinline__ int lds_byte(int r, int c) {
    const int st = (r >> 4) * 2 + (c >> 5), rr = r & 15, cc = c & 31, ob = rr * 64 + cc * 2;
    return st * 1024 + (ob ^ (((ob >> 9) & 1) << 5));
}
__device__ __forceinline__ void stage_rc(int b, int& R, int& C) {
    const int st = b / 1024, sb = b % 1024, swz = sb ^ (((sb >> 9) & 1) << 5);
    R = (st >> 1) * 16 + swz / 64; C = (st & 1) * 32 + (swz % 64) / 2;
}
__device__ __forceinline__ unsigned pack_bf16(float lo, float hi) {
    f32x2 v = {lo, hi};
    return __builtin_bit_cast(unsigned, __builtin_convertvector(v, bf16x2_t));
}
__device__ __forceinline__ __amdgpu_buffer_rsrc_t mk_rsrc(const void* p, unsigned bytes) {
    return __builtin_amdgcn_make_buffer_rsrc((void*)p, 0, (int)bytes, 0x00020000);
}
template <int AUX>
__device__ __forceinline__ void store16(const u32x4 w, const __amdgpu_buffer_rsrc_t rs, unsigned voff, unsigned soff) {
    __builtin_amdgcn_raw_buffer_store_b128(w, rs, voff, soff, AUX);
    asm volatile("s_nop 1" :: "v"(w) : "memory");
}
__device__ __forceinline__ unsigned ld_sys(const unsigned* p) { return __hip_atomic_load(p, __ATOMIC_RELAXED, __HIP_MEMORY_SCOPE_SYSTEM); }
__device__ __forceinline__ unsigned ld_agent(const unsigned* p) { return __hip_atomic_load(p, __ATOMIC_RELAXED, __HIP_MEMORY_SCOPE_AGENT); }
__device__ __forceinline__ void st_sys(unsigned* p, unsigned v) { __hip_atomic_store(p, v, __ATOMIC_RELAXED, __HIP_MEMORY_SCOPE_SYSTEM); }
__device__ __forceinline__ void st_agent(unsigned* p, unsigned v) { __hip_atomic_store(p, v, __ATOMIC_RELAXED, __HIP_MEMORY_SCOPE_AGENT); }

#define WAIT_V(n) asm volatile("s_waitcnt vmcnt(" #n ")" ::: "memory")
#define WAIT_L(n) asm volatile("s_waitcnt lgkmcnt(" #n ")" ::: "memory")
#define BAR __builtin_amdgcn_s_barrier()
#define SCHED __builtin_amdgcn_sched_barrier(0)
#define CFENCE asm volatile("" ::: "memory")

constexpr unsigned long long SEQ0 = 0x1202091101100800ull, SEQ1 = 0x14040b1913030a18ull,
                             SEQ2 = 0x16060d1b15050c1aull, SEQ3 = 0x1f1e0f1d0e17071cull;
__device__ __forceinline__ int group_code(int g) {
    const unsigned long long q = g < 8 ? SEQ0 : g < 16 ? SEQ1 : g < 24 ? SEQ2 : SEQ3;
    return (int)(q >> ((g & 7) * 8)) & 31;
}
struct Cursor { int g, pos, item; };
__device__ __forceinline__ void decode_unit(Cursor& cu, int n, int dl, int& kind, int& step, int& idx) {
    int code = group_code(cu.g);
    while (n >= cu.pos + (code < 16 ? 4 : 7)) { cu.pos += code < 16 ? 4 : 7; ++cu.g; code = group_code(cu.g); }
    const int r = n - cu.pos;
    step = code & 7;
    if (code < 16) { kind = 0; idx = (code >> 3) * 4 + ((r + ROT * dl) & 3); }
    else { kind = (code >> 3) - 1; idx = r; }
}
__device__ __forceinline__ void decode_item(Cursor& cu, int t, int& n, int& dl) {
    int code = group_code(cu.g);
    for (;;) {
        const int cnt = cu.g < 2 ? 0 : code < 16 ? 28 : 7;
        if (t < cu.item + cnt) break;
        cu.item += cnt; cu.pos += code < 16 ? 4 : 7; ++cu.g; code = group_code(cu.g);
    }
    const int r = t - cu.item;
    if (code < 16) { const int q = r / 7; n = cu.pos + q; dl = 1 + r - q * 7; }
    else { n = cu.pos + r; dl = 0; }
}
__device__ __forceinline__ unsigned src_slot(int kind, int step, int idx, int dl) {
    return kind == 0 ? (unsigned)((dl - 1) * 64 + step * 8 + idx) : (unsigned)(448 + (kind - 1) * 56 + step * 7 + idx);
}

__global__ __launch_bounds__(512, 2) void fused_gemm_rs(Args a) {
    extern __shared__ __attribute__((aligned(16))) unsigned char shm[];
    LAS unsigned char* lds = (LAS unsigned char*)shm;
    const int tid = threadIdx.x, wid = __builtin_amdgcn_readfirstlane(tid >> 6), lane = tid & 63;
    const int b = blockIdx.x, rank = a.rank & 7;
    char* const myws = a.ws[rank];
    unsigned* const tmo = (unsigned*)(myws + TMO_OFF);

    unsigned voffA, voffB;
    { int R, C; stage_rc(tid * 16, R, C); voffA = (unsigned)R * XP + (unsigned)C * 2u;
      const int row = tid >> 4, chp = tid & 15, f = ((row & 3) << 2) | ((row >> 2) & 3);
      voffB = (unsigned)row * WP + (unsigned)(chp ^ f) * 16u; }
    const unsigned voffS = (unsigned)tid * 16u;

    const bool is_copy = b < NCOPY;
    const int dd = 1 + b % 7, c = b / 7;
    const int peer = (rank + dd) & 7;
    u32x4 r[4], rn[4];
    auto load_unit = [&](int kind, int step, int idx, int dest, u32x4 (&v)[4]) {
        if (kind == 0) {
            const __amdgpu_buffer_rsrc_t rs = mk_rsrc(a.w, W_BYTES);
            const unsigned so = (unsigned)step * 64u * WP + (unsigned)(dest * 2048 + idx * 256) * 2u;
#pragma unroll
            for (int h = 0; h < 2; ++h)
#pragma unroll
                for (int i = 0; i < 2; ++i)
                    v[h * 2 + i] = __builtin_amdgcn_raw_buffer_load_b128(rs, voffB, so + h * 256u + i * 32u * WP, 0);
        } else {
            const int tm = ((rank + 1 + idx) & 7) * 2 + (kind - 1);
            const __amdgpu_buffer_rsrc_t rs = mk_rsrc(a.x, X_BYTES);
            const unsigned so = (unsigned)tm * 256u * XP + (unsigned)step * 128u;
#pragma unroll
            for (int h = 0; h < 2; ++h)
#pragma unroll
                for (int i = 0; i < 2; ++i)
                    v[h * 2 + i] = __builtin_amdgcn_raw_buffer_load_b128(rs, voffA, so + h * 128u * XP + i * 64u * XP, 0);
        }
    };
    int in = 0, idl = 0, pu = 0;
    Cursor cu = {0, 0, 0};
    if (is_copy) {
        int kind, step, idx;
        if (c < 8) { decode_item(cu, b, in, idl); decode_unit(cu, in, idl, kind, step, idx); load_unit(kind, step, idx, (rank + idl) & 7, r); }
        else { decode_unit(cu, c - 8, dd, kind, step, idx); load_unit(0, step, idx, peer, r); pu = step * 8 + idx; }
    }

    unsigned* const ctr = (unsigned*)(myws + CTR_OFF + (unsigned)b * 64u);
    const unsigned call = (unsigned)__builtin_amdgcn_readfirstlane((int)ld_agent(ctr)) + 1u;
    WAIT_V(0);
    __syncthreads();
    if (tid == 0) st_agent(ctr, call);
    const unsigned par = call & 1u;
    const __amdgpu_buffer_rsrc_t rM = mk_rsrc(myws, WS_NEED);

    int mi, nj, colrank;
    if (is_copy) {
        colrank = peer; mi = rank * 2 + (c >> 3); nj = c & 7;
        if (c < 8) {
#pragma unroll 1
            for (int k = 0;; ++k) {
                int kind, step, idx; decode_unit(cu, in, idl, kind, step, idx);
                const unsigned doff = SR_OFF + par * SR_BYTES + src_slot(kind, step, idx, idl) * UNIT;
#pragma unroll
                for (int u = 0; u < 4; ++u)
                    store16<17>(r[u], rM, voffS, doff + (unsigned)u * 8192u);
                const int fn = in, fdl = idl;
                const bool more = k + 1 < NITEMS / 56;
                if (more) { decode_item(cu, b + (k + 1) * 56, in, idl); int k2, s2, i2; decode_unit(cu, in, idl, k2, s2, i2); load_unit(k2, s2, i2, (rank + idl) & 7, rn); }
                WAIT_V(0);
                asm volatile("" : "+v"(rn[0]), "+v"(rn[1]), "+v"(rn[2]), "+v"(rn[3]));
                __syncthreads();
                if (tid == 0) {
                    const unsigned fo = SF_OFF + (unsigned)(rank * 192 + fn) * 4u;
                    if (fdl) st_sys((unsigned*)(a.ws[(rank + fdl) & 7] + fo), call);
                    else {
#pragma unroll
                        for (int p = 1; p < 8; ++p) st_sys((unsigned*)(a.ws[(rank + p) & 7] + fo), call);
                    }
                }
                if (!more) break;
#pragma unroll
                for (int u = 0; u < 4; ++u) r[u] = rn[u];
            }
        } else {
            const int cp = c - 8, dl = 8 - dd;
            const __amdgpu_buffer_rsrc_t rP = mk_rsrc(a.ws[peer], WS_NEED);
            unsigned qoff = QC_OFF + (unsigned)(dd * 8 + wid) * 64u;
            asm volatile("" : "+v"(qoff));
            unsigned* const qc = (unsigned*)(myws + qoff);
            const unsigned qbase = (call - 1u) * (unsigned)NPULL - 8u;
            const unsigned* const sf = (const unsigned*)(myws + SF_OFF) + peer * 192 + lane;
            int su = NPUSHED;
            unsigned f0, f1, f2;
            auto sf_issue = [&]() { f0 = ld_sys(sf); f1 = ld_sys(sf + 64); f2 = ld_sys(sf + 128); };
            auto sf_eval = [&]() {
                const unsigned long long m0 = ~__builtin_amdgcn_ballot_w64((int)(f0 - call) >= 0 || lane < NPUSHED);
                const unsigned long long m1 = ~__builtin_amdgcn_ballot_w64((int)(f1 - call) >= 0);
                const unsigned long long m2 = ~__builtin_amdgcn_ballot_w64((int)(f2 - call) >= 0 || lane >= NUNITS - 128);
                su = m0 ? __builtin_ctzll(m0) : m1 ? 64 + __builtin_ctzll(m1) : m2 ? 128 + __builtin_ctzll(m2) : 192;
            };
            auto sf_wait = [&](int n) {
                if (n < su) return;
                const unsigned long long t0 = __builtin_amdgcn_s_memrealtime();
                for (;;) {
                    sf_issue(); sf_eval();
                    if (n < su) break;
                    if ((unsigned)__builtin_amdgcn_readfirstlane((int)ld_agent(tmo)) == call ||
                        __builtin_amdgcn_s_memrealtime() - t0 > DEADLINE_TICKS) {
                        if (lane == 0) st_agent(tmo, call);
                        su = 192; break;
                    }
                    __builtin_amdgcn_s_sleep(2);
                }
            };
            auto unit = [&](int q, int& n, unsigned& so, unsigned& dsto, unsigned& flo) {
                n = NPUSHED + q;
                int kind, step, idx; decode_unit(cu, n, dl, kind, step, idx);
                so = SR_OFF + par * SR_BYTES + src_slot(kind, step, idx, dl) * UNIT;
                if (kind == 0) {
                    dsto = WD_OFF + par * WD_BYTES + (unsigned)((peer * 8 + step) * 8 + idx) * UNIT;
                    flo = WF_OFF + (unsigned)(idx * 64 + peer * 8 + step) * 16u + (unsigned)wid * 2u;
                } else {
                    const int tm = ((peer + 1 + idx) & 7) * 2 + (kind - 1);
                    dsto = XD_OFF + (unsigned)((peer * 8 + step) * 16 + tm) * UNIT;
                    flo = XF_OFF + (unsigned)(tm * 64 + peer * 8 + step) * 16u + (unsigned)wid * 2u;
                }
            };
            auto set_flag = [&](const __amdgpu_buffer_rsrc_t rs, unsigned flo) {
                if (lane == 0) __builtin_amdgcn_raw_buffer_store_b16((unsigned short)call, rs, 0, flo, 17);
            };
            const unsigned poff = WD_OFF + par * WD_BYTES + (unsigned)(rank * 64 + pu) * UNIT;
            store16<17>(r[0], rP, voffS, poff); WAIT_V(0);
            store16<17>(r[1], rP, voffS, poff + 8192u); WAIT_V(0);
            store16<17>(r[2], rP, voffS, poff + 16384u); sf_issue(); WAIT_V(0);
            store16<17>(r[3], rP, voffS, poff + 24576u);
            sf_eval();
            int n0, n1 = 0; unsigned so0, do0, fl0, so1 = 0, do1 = 0, fl1 = 0, flp = 0;
            unit(cp, n0, so0, do0, fl0);
            sf_wait(n0);
            u32x4 v = __builtin_amdgcn_raw_buffer_load_b128(rP, voffS, so0, 17);
            bool pend = false;
            unsigned qv = 0;
#pragma unroll 1
            for (int k = 0;; ++k) {
                bool more = false;
#pragma unroll
                for (int u = 0; u < 4; ++u) {
                    WAIT_V(0);
                    asm volatile("" : "+v"(v));
                    const u32x4 w = v;
                    if (u < 3) v = __builtin_amdgcn_raw_buffer_load_b128(rP, voffS, so0 + (unsigned)(u + 1) * 8192u, 17);
                    else if (more) {
                        if (pend) { sf_eval(); pend = false; }
                        sf_wait(n1);
                        v = __builtin_amdgcn_raw_buffer_load_b128(rP, voffS, so1, 17);
                        if (su < NUNITS) { sf_issue(); pend = true; }
                    }
                    store16<17>(w, rM, voffS, do0 + (unsigned)u * 8192u);
                    if (u == 0) {
                        if (k == 0) { set_flag(rP, WF_OFF + (unsigned)((pu & 7) * 64 + rank * 8 + (pu >> 3)) * 16u + (unsigned)wid * 2u); }
                        else set_flag(rM, flp);
                        if (lane == 0) qv = __hip_atomic_fetch_add(qc, 1u, __ATOMIC_RELAXED, __HIP_MEMORY_SCOPE_AGENT);
                    }
                    if (u == 2) {
                        const int qn = __builtin_amdgcn_readfirstlane((int)(qv - qbase));
                        more = qn < NPULL;
                        if (more) unit(qn, n1, so1, do1, fl1);
                    }
                }
                flp = fl0;
                if (!more) break;
                so0 = so1; do0 = do1; fl0 = fl1;
            }
            WAIT_V(0);
            set_flag(rM, flp);
        }
    } else {
        const int g = b - NCOPY, xcd = g & 7, idx = g >> 3;
        mi = 4 * (xcd >> 1) + (idx & 3); nj = 4 * (xcd & 1) + (idx >> 2); colrank = rank;
    }
    const int rb = mi >> 1, hh = mi & 1;

    const int wr = wid >> 2, wc = wid & 3, fr = lane & 15, fq = lane >> 4;
    const unsigned ldsw = (unsigned)wid * 1024u;
    const int aoff = lds_byte(wr * 64 + fr, fq * 8);
    unsigned boff[2][2];
    { const int q = fr >> 2, p = lane & 3;
#pragma unroll
      for (int n = 0; n < 2; ++n)
#pragma unroll
          for (int t = 0; t < 2; ++t) {
              const int ch = 4 * wc + 2 * n + (p >> 1), f = (q << 2) | ((2 * fq + t) & 3);
              boff[n][t] = (unsigned)(size_t)lds + 65536u + 256u * (unsigned)(8 * fq + 4 * t + q) + 16u * (unsigned)(ch ^ f) + 8u * (unsigned)(p & 1);
          } }
#define SA(bb, h) (((bb) * 2 + (h)) * HTB)
#define SB(bb, h) ((4 + (bb) * 2 + (h)) * HTB)
#define SBR(bb, h) (((bb) * 2 + (h)) * HTB)
#define STAGE(bufoff, rs, voff, soff, istr) do { _Pragma("unroll") for (int _i = 0; _i < 2; ++_i) \
        __builtin_amdgcn_raw_ptr_buffer_load_lds(rs, (LAS void*)(lds + (bufoff) + ldsw + _i * 8192), 16, voff, (soff) + _i * (istr), 0, 16); } while (0)
#define LDA(dst, bb, h) do { _Pragma("unroll") for (int m = 0; m < 4; ++m) _Pragma("unroll") for (int k = 0; k < 2; ++k) \
        dst[m][k] = *(const LAS bf16x8*)(lds + SA(bb, h) + aoff + m * 2048 + k * 1024); } while (0)
#define TRR(d, addr, off) asm volatile("ds_read_b64_tr_b16 %0, %1 offset:%2" : "=&v"(d) : "v"(addr), "i"(off) : "memory")
#define LDB(dst, bb, h) do { _Pragma("unroll") for (int n = 0; n < 2; ++n) _Pragma("unroll") for (int k = 0; k < 2; ++k) { \
        TRR(dst[n][k][0], boff[n][0], SBR(bb, h) + k * 8192); TRR(dst[n][k][1], boff[n][1], SBR(bb, h) + k * 8192); } } while (0)
#define PKB(v) __builtin_shufflevector((v)[0], (v)[1], 0, 1, 2, 3, 4, 5, 6, 7)
#define MMA(ai, bj, At, Bt) do { SCHED; __builtin_amdgcn_s_setprio(1); _Pragma("unroll") for (int m = 0; m < 4; ++m) _Pragma("unroll") for (int n = 0; n < 2; ++n) \
        _Pragma("unroll") for (int k = 0; k < 2; ++k) acc[ai][bj][m][n] = __builtin_amdgcn_mfma_f32_16x16x32_bf16(PKB(Bt[n][k]), At[m][k], acc[ai][bj][m][n], 0, 0, 0); \
        __builtin_amdgcn_s_setprio(0); } while (0)

    f32x4 acc[2][2][4][2];
#pragma unroll
    for (int i0 = 0; i0 < 2; ++i0)
#pragma unroll
        for (int i1 = 0; i1 < 2; ++i1)
#pragma unroll
            for (int m = 0; m < 4; ++m)
#pragma unroll
                for (int n = 0; n < 2; ++n) acc[i0][i1][m][n] = (f32x4){0.f, 0.f, 0.f, 0.f};
    bf16x8 At[4][2]; s16x4 B0[2][2][2], B1[2][2][2];

    const int np = (rb == rank) ? 7 : 6;
    const unsigned long long valid = ~((0xFFull << (rank * 8)) | (0xFFull << (rb * 8)));
    const int nph = !is_copy || c < 8 ? 2 : 0;
    const unsigned dgo = DG_OFF + (unsigned)(b % 56) * 262144u + (unsigned)wid * 32768u;
    unsigned* const dgf = (unsigned*)(myws + DF_OFF) + (b % 56) * 8 + wid;
#pragma unroll 1
    for (int ph = 0; ph < nph; ++ph) {
        const bool own = is_copy || ph == 0;
        const int mic = is_copy ? rank * 2 + 1 - ph : mi;
        const int nt = own ? 8 : np * 8;
        const __amdgpu_buffer_rsrc_t rA = mk_rsrc(own ? a.x : (const char*)myws, own ? X_BYTES : WS_NEED);
        const __amdgpu_buffer_rsrc_t rB = mk_rsrc(own ? a.w : (const char*)myws, own ? W_BYTES : WS_NEED);
        const unsigned vA = own ? voffA : voffS, vB = own ? voffB : voffS;
        const unsigned iA = own ? 64u * XP : 8192u, iB = own ? 32u * WP : 8192u;
        const unsigned hA = own ? 128u * XP : HTB, hB = own ? 256u : HTB;
        auto addrA = [&](int id) -> unsigned {
            return own ? (unsigned)mic * 256u * XP + (unsigned)id * 128u
                       : XD_OFF + (unsigned)(id * 16 + mi) * UNIT;
        };
        auto addrB = [&](int id) -> unsigned {
            return own ? (unsigned)id * 64u * WP + (unsigned)(colrank * 2048 + nj * 256) * 2u
                       : WD_OFF + par * WD_BYTES + (unsigned)(id * 8 + nj) * UNIT;
        };
        unsigned long long ready = own ? 0xFFull : 0ull, seen = ready;
        const unsigned xf = XF_OFF + (unsigned)(mi * 64) * 16u, wf = WF_OFF + (unsigned)(nj * 64) * 16u;
        LAS volatile unsigned* const lavail = (LAS volatile unsigned*)(lds + 131072);
        int pslot = 0;
        const short c16 = (short)call;
        const s16x8 c8 = {c16, c16, c16, c16, c16, c16, c16, c16};
        auto arrived = [&](const u32x4 fx, const u32x4 fw) -> unsigned long long {
            const s16x8 dx = __builtin_bit_cast(s16x8, fx) - c8, dw = __builtin_bit_cast(s16x8, fw) - c8;
            const u32x4 dd4 = __builtin_bit_cast(u32x4, dx) | __builtin_bit_cast(u32x4, dw);
            const bool ok = ((dd4.x | dd4.y | dd4.z | dd4.w) & 0x80008000u) == 0u;
            return __builtin_amdgcn_ballot_w64(ok) & valid & ~seen;
        };
        auto pop = [&]() -> int { const int id = __builtin_ctzll(ready); ready &= ready - 1ull; return id; };
        int remaining = nt;
#pragma unroll 1
        while (remaining > 0) {
#pragma unroll 1
            while (__builtin_popcountll(ready) < 2) {
                if (wid == 0) {
                    const unsigned long long t0 = __builtin_amdgcn_s_memrealtime();
                    unsigned long long mask;
                    for (;;) {
                        CFENCE;
                        const u32x4 fx = __builtin_amdgcn_raw_buffer_load_b128(rM, (unsigned)lane * 16u, xf, 17);
                        const u32x4 fw = __builtin_amdgcn_raw_buffer_load_b128(rM, (unsigned)lane * 16u, wf, 17);
                        mask = arrived(fx, fw);
                        if (mask != 0ull) break;
                        if ((unsigned)__builtin_amdgcn_readfirstlane((int)ld_agent(tmo)) == call ||
                            __builtin_amdgcn_s_memrealtime() - t0 > DEADLINE_TICKS) {
                            if (lane == 0) st_agent(tmo, call);
                            mask = valid & ~seen; break;
                        }
                        __builtin_amdgcn_s_sleep(2);
                    }
                    if (lane == 0) { lavail[pslot] = (unsigned)mask; lavail[pslot + 1] = (unsigned)(mask >> 32); }
                    WAIT_V(0); WAIT_L(0);
                }
                CFENCE; BAR; CFENCE;
                const unsigned lo = (unsigned)__builtin_amdgcn_readfirstlane((int)lavail[pslot]), hi = (unsigned)__builtin_amdgcn_readfirstlane((int)lavail[pslot + 1]);
                pslot ^= 2;
                const unsigned long long m = ((unsigned long long)hi << 32) | lo;
                ready |= m; seen |= m;
            }
            const int id0 = pop(), id1 = pop();
            { const unsigned a0 = addrA(id0), b0 = addrB(id0), a1 = addrA(id1), b1 = addrB(id1);
              STAGE(SB(0, 0), rB, vB, b0, iB); STAGE(SA(0, 0), rA, vA, a0, iA); STAGE(SB(0, 1), rB, vB, b0 + hB, iB); STAGE(SA(0, 1), rA, vA, a0 + hA, iA);
              if (wr == 1) BAR;
              WAIT_V(4); BAR;
              STAGE(SB(1, 0), rB, vB, b1, iB); STAGE(SA(1, 0), rA, vA, a1, iA); STAGE(SB(1, 1), rB, vB, b1 + hB, iB);
              WAIT_V(6); BAR; }
            unsigned a1 = addrA(id1);
#pragma unroll 1
            for (;;) {
                remaining -= 2;
                const bool last = __builtin_popcountll(ready) < 2;
                const int t2 = last ? id0 : pop();
                const int t3 = last ? id0 : pop();
                const bool look = !own && (valid & ~seen) != 0ull;
                const unsigned a2 = addrA(t2), b2 = addrB(t2);
                const unsigned a3 = addrA(t3), b3 = addrB(t3);
                u32x4 fx, fw;
                LDB(B0, 0, 0); SCHED; LDA(At, 0, 0); STAGE(SA(1, 1), rA, vA, a1 + hA, iA);
                WAIT_L(8); BAR; WAIT_L(0); MMA(0, 0, At, B0); BAR; SCHED;
                LDB(B1, 0, 1); STAGE(SB(0, 0), rB, vB, b2, iB);
                BAR; WAIT_L(0); MMA(0, 1, At, B1); BAR;
                LDA(At, 0, 1); STAGE(SA(0, 0), rA, vA, a2, iA);
                BAR; WAIT_L(0); MMA(1, 0, At, B0); BAR; SCHED;
                STAGE(SB(0, 1), rB, vB, b2 + hB, iB);
                WAIT_V(6); BAR; MMA(1, 1, At, B1); BAR;
                if (look && wid == 0) {
                    fx = __builtin_amdgcn_raw_buffer_load_b128(rM, (unsigned)lane * 16u, xf, 17);
                    fw = __builtin_amdgcn_raw_buffer_load_b128(rM, (unsigned)lane * 16u, wf, 17);
                }
                LDB(B0, 1, 0); SCHED; LDA(At, 1, 0); STAGE(SA(0, 1), rA, vA, a2 + hA, iA);
                WAIT_L(8); BAR; WAIT_L(0); MMA(0, 0, At, B0); BAR; SCHED;
                LDB(B1, 1, 1); STAGE(SB(1, 0), rB, vB, b3, iB);
                BAR; WAIT_L(0); MMA(0, 1, At, B1); BAR;
                LDA(At, 1, 1); STAGE(SA(1, 0), rA, vA, a3, iA);
                BAR; WAIT_L(0); MMA(1, 0, At, B0); BAR; SCHED;
                STAGE(SB(1, 1), rB, vB, b3 + hB, iB);
                WAIT_V(6);
                if (look && wid == 0) {
                    const unsigned long long mask = arrived(fx, fw);
                    if (lane == 0) { lavail[4] = (unsigned)mask; lavail[5] = (unsigned)(mask >> 32); }
                    WAIT_L(0);
                }
                CFENCE; BAR; CFENCE;
                unsigned lo = 0, hi = 0;
                if (look) { lo = lavail[4]; hi = lavail[5]; }
                MMA(1, 1, At, B1); BAR;
                if (look) {
                    const unsigned long long m = ((unsigned long long)(unsigned)__builtin_amdgcn_readfirstlane((int)hi) << 32) | (unsigned)__builtin_amdgcn_readfirstlane((int)lo);
                    ready |= m; seen |= m;
                }
                a1 = a3;
                if (last) break;
            }
            WAIT_V(0);
            if (wr == 0) BAR;
            BAR;
        }
        if (is_copy && ph == 0) {
#pragma unroll
            for (int u = 0; u < 32; ++u) {
                f32x4& v = acc[u >> 4][(u >> 3) & 1][(u >> 1) & 3][u & 1];
                store16<17>(__builtin_bit_cast(u32x4, v), rM, (unsigned)lane * 16u, dgo + (unsigned)u * 1024u);
                v = (f32x4){0.f, 0.f, 0.f, 0.f};
            }
            WAIT_V(0);
            if (lane == 0) st_agent(dgf, call);
        }
    }
    if (nph == 0) {
        const unsigned long long t0 = __builtin_amdgcn_s_memrealtime();
        while ((unsigned)__builtin_amdgcn_readfirstlane((int)ld_agent(dgf)) != call) {
            if ((unsigned)__builtin_amdgcn_readfirstlane((int)ld_agent(tmo)) == call ||
                __builtin_amdgcn_s_memrealtime() - t0 > DEADLINE_TICKS) {
                if (lane == 0) st_agent(tmo, call);
                break;
            }
            __builtin_amdgcn_s_sleep(2);
        }
        CFENCE;
#pragma unroll
        for (int u = 0; u < 32; ++u)
            acc[u >> 4][(u >> 3) & 1][(u >> 1) & 3][u & 1] =
                __builtin_bit_cast(f32x4, __builtin_amdgcn_raw_buffer_load_b128(rM, (unsigned)lane * 16u, dgo + (unsigned)u * 1024u, 17));
    }

    const bool bad0 = (unsigned)__builtin_amdgcn_readfirstlane((int)ld_agent(tmo)) == call;
    const u32x4 nan4 = {0x7FC07FC0u, 0x7FC07FC0u, 0x7FC07FC0u, 0x7FC07FC0u};
    constexpr unsigned PSTR = 8192u;
    const unsigned pbase = PS_OFF + par * PS_BYTES + (unsigned)(mi * 8 + nj) * 131072u + (unsigned)wid * 1024u;
    if (!is_copy && rb != rank) {
#pragma unroll
        for (int ai = 0; ai < 2; ++ai)
#pragma unroll
            for (int bj = 0; bj < 2; ++bj)
#pragma unroll
                for (int m = 0; m < 4; ++m) {
                    const f32x4 v0 = acc[ai][bj][m][0], v1 = acc[ai][bj][m][1];
                    u32x4 w; w.x = pack_bf16(v0[0], v0[1]); w.y = pack_bf16(v0[2], v0[3]); w.z = pack_bf16(v1[0], v1[1]); w.w = pack_bf16(v1[2], v1[3]);
                    if (bad0) w = nan4;
                    store16<17>(w, rM, (unsigned)lane * 16u, pbase + (unsigned)((ai * 2 + bj) * 4 + m) * PSTR);
                }
        WAIT_V(0);
        if (lane == 0) st_sys((unsigned*)(a.ws[rb] + PF_OFF) + (unsigned)((rank * 2 + hh) * 8 + nj) * 32u + wid, call);
        return;
    }
    const __amdgpu_buffer_rsrc_t rO = mk_rsrc(a.out, O_BYTES);
    const unsigned vo = (unsigned)(wr * 64 + fr) * WP + (unsigned)(wc * 32 + ((fq & 1) << 4) + ((fq >> 1) << 3)) * 2u;
    const unsigned so = (unsigned)hh * 256u * WP + (unsigned)(colrank * 2048 + nj * 256) * 2u;
    auto store_out = [&](int ai, int bj, int m, bool bad) {
        const f32x4 v0 = acc[ai][bj][m][0], v1 = acc[ai][bj][m][1];
        const unsigned ax = pack_bf16(v0[0], v0[1]), ay = pack_bf16(v0[2], v0[3]);
        const unsigned bx = pack_bf16(v1[0], v1[1]), by = pack_bf16(v1[2], v1[3]);
        const auto sx = __builtin_amdgcn_permlane16_swap(ax, bx, false, false);
        const auto sy = __builtin_amdgcn_permlane16_swap(ay, by, false, false);
        u32x4 w = {sx[0], sy[0], sx[1], sy[1]};
        if (bad) w = nan4;
        store16<OUT_AUX>(w, rO, vo, so + (unsigned)(ai * 128 + m * 16) * WP + (unsigned)bj * 256u);
    };
    if (is_copy) {
        const unsigned* const pf = (const unsigned*)(myws + PF_OFF) + (unsigned)((colrank * 2 + hh) * 8 + nj) * 32u + wid;
        const unsigned long long t0 = __builtin_amdgcn_s_memrealtime();
        bool bad = bad0;
        while ((int)((unsigned)__builtin_amdgcn_readfirstlane((int)ld_sys(pf)) - call) < 0) {
            if ((unsigned)__builtin_amdgcn_readfirstlane((int)ld_agent(tmo)) == call ||
                __builtin_amdgcn_s_memrealtime() - t0 > DEADLINE_TICKS) {
                if (lane == 0) st_agent(tmo, call);
                bad = true;
                break;
            }
            __builtin_amdgcn_s_sleep(2);
        }
        CFENCE;
        const __amdgpu_buffer_rsrc_t rP = mk_rsrc(a.ws[colrank], WS_NEED);
        auto combine = [&](int u, const u32x4 w) {
            const int ai = u >> 3, bj = (u >> 2) & 1, m = u & 3;
            f32x4& v0 = acc[ai][bj][m][0]; f32x4& v1 = acc[ai][bj][m][1];
            v0[0] += __uint_as_float(w.x << 16); v0[1] += __uint_as_float(w.x & 0xFFFF0000u);
            v0[2] += __uint_as_float(w.y << 16); v0[3] += __uint_as_float(w.y & 0xFFFF0000u);
            v1[0] += __uint_as_float(w.z << 16); v1[1] += __uint_as_float(w.z & 0xFFFF0000u);
            v1[2] += __uint_as_float(w.w << 16); v1[3] += __uint_as_float(w.w & 0xFFFF0000u);
            store_out(ai, bj, m, bad);
        };
        auto pull = [&](auto winc) {
            constexpr int W = decltype(winc)::value;
            u32x4 pv[W];
#pragma unroll
            for (int j = 0; j < W; ++j) pv[j] = __builtin_amdgcn_raw_buffer_load_b128(rP, (unsigned)lane * 16u, pbase + (unsigned)j * PSTR, 17);
#pragma unroll
            for (int u = 0; u < 16; ++u) {
                const int nl = (u + W - 1 < 16 ? W - 1 : 15 - u), ns = u < W ? u : W;
                switch (nl + ns) {
                    case 0: WAIT_V(0); break; case 1: WAIT_V(1); break; case 2: WAIT_V(2); break; case 3: WAIT_V(3); break;
                    case 4: WAIT_V(4); break; case 5: WAIT_V(5); break; case 6: WAIT_V(6); break; default: WAIT_V(7); break;
                }
                asm volatile("" : "+v"(pv[u % W]));
                const u32x4 w = pv[u % W];
                if (u + W < 16) pv[u % W] = __builtin_amdgcn_raw_buffer_load_b128(rP, (unsigned)lane * 16u, pbase + (unsigned)(u + W) * PSTR, 17);
                SCHED;
                combine(u, w);
                SCHED;
            }
        };
        if (hh == 0) pull(std::integral_constant<int, WINU>{}); else pull(std::integral_constant<int, WINL>{});
    } else {
#pragma unroll
        for (int ai = 0; ai < 2; ++ai)
#pragma unroll
            for (int m = 0; m < 4; ++m)
#pragma unroll
                for (int bj = 0; bj < 2; ++bj) store_out(ai, bj, m, bad0);
    }
}

}

extern "C" void kernel_launch(int rank, int nranks, ncclComm_t comm,
                              hipStream_t stream,
                              void* const* d_in, const int* in_sizes, int n_in,
                              void* d_out, int out_size,
                              const int* in_dtypes, int out_dtype,
                              void* const* d_out_all,
                              void* const* d_ws_all, size_t ws_bytes) {
    (void)comm; (void)in_dtypes; (void)out_dtype; (void)d_out_all;
    if (nranks != NR || n_in < 2 || rank < 0 || rank >= NR || ws_bytes < (size_t)WS_NEED ||
        in_sizes[0] != 4096 * 512 || in_sizes[1] != 512 * 16384 || out_size != 512 * 16384) {
        fprintf(stderr, "kernel_launch: unexpected problem shape\n");
        return;
    }
    static bool attr_set[64] = {};
    int dev = 0;
    HIP_CHECK(hipGetDevice(&dev));
    if (dev >= 0 && dev < 64 && !attr_set[dev]) {
        HIP_CHECK(hipFuncSetAttribute((const void*)fused_gemm_rs, hipFuncAttributeMaxDynamicSharedMemorySize, (int)LDS_BYTES));
        attr_set[dev] = true;
    }
    Args a;
    a.x = (const char*)d_in[0];
    a.w = (const char*)d_in[1];
    a.out = (char*)d_out;
    for (int p = 0; p < NR; ++p) a.ws[p] = (char*)d_ws_all[p];
    a.rank = rank;
    hipLaunchKernelGGL(fused_gemm_rs, dim3(NBLK), dim3(512), LDS_BYTES, stream, a);
    HIP_CHECK(hipGetLastError());
}
```

```cpp
#include <hip/hip_runtime.h>
#include <rccl/rccl.h>
#include <cstdio>
#include <cstdint>
#include <type_traits>

#define HIP_CHECK(expr) do { hipError_t _e = (expr); if (_e != hipSuccess) { \
    fprintf(stderr, "HIP error %s at %s:%d\n", hipGetErrorString(_e), __FILE__, __LINE__); \
    return; } } while (0)

namespace {

#define LAS __attribute__((address_space(3)))
typedef short bf16x8 __attribute__((ext_vector_type(8)));
typedef short s16x4 __attribute__((ext_vector_type(4)));
typedef short s16x8 __attribute__((ext_vector_type(8)));
typedef float f32x4 __attribute__((ext_vector_type(4)));
typedef float f32x2 __attribute__((ext_vector_type(2)));
typedef unsigned u32x4 __attribute__((ext_vector_type(4)));
typedef __bf16 bf16x2_t __attribute__((ext_vector_type(2)));

constexpr int NR = 8;
constexpr unsigned XP = 1024;
constexpr unsigned WP = 32768;
constexpr unsigned X_BYTES = 4096u * XP;
constexpr unsigned W_BYTES = 512u * WP;
constexpr unsigned O_BYTES = 512u * WP;
constexpr unsigned UNIT = 32768;
constexpr unsigned HTB = 16384;

constexpr unsigned CTR_OFF = 0;
constexpr unsigned TMO_OFF = 16384;
constexpr unsigned QC_OFF = 20480;
constexpr unsigned SF_OFF = 24576;
constexpr unsigned DF_OFF = 30720;
constexpr unsigned PF_OFF = 32768;
constexpr unsigned XF_OFF = 65536;
constexpr unsigned WF_OFF = 81920;
constexpr unsigned XD_OFF = 1u << 20;
constexpr unsigned WD_OFF = 33u << 20, WD_BYTES = 16u << 20;
constexpr unsigned SR_OFF = 65u << 20, SR_BYTES = 18u << 20;
constexpr unsigned PS_OFF = 101u << 20, PS_BYTES = 16u << 20;
constexpr unsigned DG_OFF = PS_OFF + 2 * PS_BYTES;
constexpr unsigned WS_NEED = DG_OFF + 56u * 262144u;

constexpr int NCOPY = 112, NGEMM = 128, NBLK = NCOPY + NGEMM;
constexpr int OUT_AUX = 17;
constexpr int WINU = 1, WINL = 1;
constexpr int NPW = 7;
constexpr int ROT = 1;
constexpr int NUNITS = 176;
constexpr int NPUSHED = 8;
constexpr int NPULL = NUNITS - NPUSHED;
constexpr int NITEMS = 504;
constexpr unsigned LDS_BYTES = 131072 + 64;
constexpr unsigned long long DEADLINE_TICKS = 40000000ull;

struct Args {
    const char* x; const char* w; char* out;
    char* ws[NR];
    int rank;
};

__device__ __forceinline__ int lds_byte(int r, int c) {
    const int st = (r >> 4) * 2 + (c >> 5), rr = r & 15, cc = c & 31, ob = rr * 64 + cc * 2;
    return st * 1024 + (ob ^ (((ob >> 9) & 1) << 5));
}
__device__ __forceinline__ void stage_rc(int b, int& R, int& C) {
    const int st = b / 1024, sb = b % 1024, swz = sb ^ (((sb >> 9) & 1) << 5);
    R = (st >> 1) * 16 + swz / 64; C = (st & 1) * 32 + (swz % 64) / 2;
}
__device__ __forceinline__ unsigned pack_bf16(float lo, float hi) {
    f32x2 v = {lo, hi};
    return __builtin_bit_cast(unsigned, __builtin_convertvector(v, bf16x2_t));
}
__device__ __forceinline__ __amdgpu_buffer_rsrc_t mk_rsrc(const void* p, unsigned bytes) {
    return __builtin_amdgcn_make_buffer_rsrc((void*)p, 0, (int)bytes, 0x00020000);
}
template <int AUX>
__device__ __forceinline__ void store16(const u32x4 w, const __amdgpu_buffer_rsrc_t rs, unsigned voff, unsigned soff) {
    __builtin_amdgcn_raw_buffer_store_b128(w, rs, voff, soff, AUX);
    asm volatile("s_nop 1" :: "v"(w) : "memory");
}
__device__ __forceinline__ unsigned ld_sys(const unsigned* p) { return __hip_atomic_load(p, __ATOMIC_RELAXED, __HIP_MEMORY_SCOPE_SYSTEM); }
__device__ __forceinline__ unsigned ld_agent(const unsigned* p) { return __hip_atomic_load(p, __ATOMIC_RELAXED, __HIP_MEMORY_SCOPE_AGENT); }
__device__ __forceinline__ void st_sys(unsigned* p, unsigned v) { __hip_atomic_store(p, v, __ATOMIC_RELAXED, __HIP_MEMORY_SCOPE_SYSTEM); }
__device__ __forceinline__ void st_agent(unsigned* p, unsigned v) { __hip_atomic_store(p, v, __ATOMIC_RELAXED, __HIP_MEMORY_SCOPE_AGENT); }

#define WAIT_V(n) asm volatile("s_waitcnt vmcnt(" #n ")" ::: "memory")
#define WAIT_L(n) asm volatile("s_waitcnt lgkmcnt(" #n ")" ::: "memory")
#define BAR __builtin_amdgcn_s_barrier()
#define SCHED __builtin_amdgcn_sched_barrier(0)
#define CFENCE asm volatile("" ::: "memory")

constexpr unsigned long long SEQ0 = 0x1202091101100800ull, SEQ1 = 0x14040b1913030a18ull,
                             SEQ2 = 0x16060d1b15050c1aull, SEQ3 = 0x1f1e0f1d0e17071cull;
__device__ __forceinline__ int group_code(int g) {
    const unsigned long long q = g < 8 ? SEQ0 : g < 16 ? SEQ1 : g < 24 ? SEQ2 : SEQ3;
    return (int)(q >> ((g & 7) * 8)) & 31;
}
struct Cursor { int g, pos, item; };
__device__ __forceinline__ void decode_unit(Cursor& cu, int n, int dl, int& kind, int& step, int& idx) {
    int code = group_code(cu.g);
    while (n >= cu.pos + (code < 16 ? 4 : 7)) { cu.pos += code < 16 ? 4 : 7; ++cu.g; code = group_code(cu.g); }
    const int r = n - cu.pos;
    step = code & 7;
    if (code < 16) { kind = 0; idx = (code >> 3) * 4 + ((r + ROT * dl) & 3); }
    else { kind = (code >> 3) - 1; idx = r; }
}
__device__ __forceinline__ void decode_item(Cursor& cu, int t, int& n, int& dl) {
    int code = group_code(cu.g);
    for (;;) {
        const int cnt = cu.g < 2 ? 0 : code < 16 ? 28 : 7;
        if (t < cu.item + cnt) break;
        cu.item += cnt; cu.pos += code < 16 ? 4 : 7; ++cu.g; code = group_code(cu.g);
    }
    const int r = t - cu.item;
    if (code < 16) { const int q = r / 7; n = cu.pos + q; dl = 1 + r - q * 7; }
    else { n = cu.pos + r; dl = 0; }
}
__device__ __forceinline__ unsigned src_slot(int kind, int step, int idx, int dl) {
    return kind == 0 ? (unsigned)((dl - 1) * 64 + step * 8 + idx) : (unsigned)(448 + (kind - 1) * 56 + step * 7 + idx);
}

__global__ __launch_bounds__(512, 2) void fused_gemm_rs(Args a) {
    extern __shared__ __attribute__((aligned(16))) unsigned char shm[];
    LAS unsigned char* lds = (LAS unsigned char*)shm;
    const int tid = threadIdx.x, wid = __builtin_amdgcn_readfirstlane(tid >> 6), lane = tid & 63;
    const int b = blockIdx.x, rank = a.rank & 7;
    char* const myws = a.ws[rank];
    unsigned* const tmo = (unsigned*)(myws + TMO_OFF);

    unsigned voffA, voffB;
    { int R, C; stage_rc(tid * 16, R, C); voffA = (unsigned)R * XP + (unsigned)C * 2u;
      const int row = tid >> 4, chp = tid & 15, f = ((row & 3) << 2) | ((row >> 2) & 3);
      voffB = (unsigned)row * WP + (unsigned)(chp ^ f) * 16u; }
    const unsigned voffS = (unsigned)tid * 16u;

    const bool is_copy = b < NCOPY;
    const int dd = 1 + b % 7, c = b / 7;
    const int peer = (rank + dd) & 7;
    u32x4 r[4], rn[4];
    auto load_unit = [&](int kind, int step, int idx, int dest, u32x4 (&v)[4]) {
        if (kind == 0) {
            const __amdgpu_buffer_rsrc_t rs = mk_rsrc(a.w, W_BYTES);
            const unsigned so = (unsigned)step * 64u * WP + (unsigned)(dest * 2048 + idx * 256) * 2u;
#pragma unroll
            for (int h = 0; h < 2; ++h)
#pragma unroll
                for (int i = 0; i < 2; ++i)
                    v[h * 2 + i] = __builtin_amdgcn_raw_buffer_load_b128(rs, voffB, so + h * 256u + i * 32u * WP, 0);
        } else {
            const int tm = ((rank + 1 + idx) & 7) * 2 + (kind - 1);
            const __amdgpu_buffer_rsrc_t rs = mk_rsrc(a.x, X_BYTES);
            const unsigned so = (unsigned)tm * 256u * XP + (unsigned)step * 128u;
#pragma unroll
            for (int h = 0; h < 2; ++h)
#pragma unroll
                for (int i = 0; i < 2; ++i)
                    v[h * 2 + i] = __builtin_amdgcn_raw_buffer_load_b128(rs, voffA, so + h * 128u * XP + i * 64u * XP, 0);
        }
    };
    int in = 0, idl = 0, pu = 0;
    Cursor cu = {0, 0, 0};
    if (is_copy) {
        int kind, step, idx;
        if (c < 8) { decode_item(cu, b, in, idl); decode_unit(cu, in, idl, kind, step, idx); load_unit(kind, step, idx, (rank + idl) & 7, r); }
        else { decode_unit(cu, c - 8, dd, kind, step, idx); load_unit(0, step, idx, peer, r); pu = step * 8 + idx; }
    }

    unsigned* const ctr = (unsigned*)(myws + CTR_OFF + (unsigned)b * 64u);
    const unsigned call = (unsigned)__builtin_amdgcn_readfirstlane((int)ld_agent(ctr)) + 1u;
    WAIT_V(0);
    __syncthreads();
    if (tid == 0) st_agent(ctr, call);
    const unsigned par = call & 1u;
    const __amdgpu_buffer_rsrc_t rM = mk_rsrc(myws, WS_NEED);

    int mi, nj, colrank;
    if (is_copy) {
        colrank = peer; mi = rank * 2 + (c >> 3); nj = c & 7;
        if (c < 8) {
#pragma unroll 1
            for (int k = 0;; ++k) {
                int kind, step, idx; decode_unit(cu, in, idl, kind, step, idx);
                const unsigned doff = SR_OFF + par * SR_BYTES + src_slot(kind, step, idx, idl) * UNIT;
#pragma unroll
                for (int u = 0; u < 4; ++u)
                    store16<17>(r[u], rM, voffS, doff + (unsigned)u * 8192u);
                const int fn = in, fdl = idl;
                const bool more = k + 1 < NITEMS / 56;
                if (more) { decode_item(cu, b + (k + 1) * 56, in, idl); int k2, s2, i2; decode_unit(cu, in, idl, k2, s2, i2); load_unit(k2, s2, i2, (rank + idl) & 7, rn); }
                WAIT_V(0);
                asm volatile("" : "+v"(rn[0]), "+v"(rn[1]), "+v"(rn[2]), "+v"(rn[3]));
                __syncthreads();
                if (tid == 0) {
                    const unsigned fo = SF_OFF + (unsigned)(rank * 192 + fn) * 4u;
                    if (fdl) st_sys((unsigned*)(a.ws[(rank + fdl) & 7] + fo), call);
                    else {
#pragma unroll
                        for (int p = 1; p < 8; ++p) st_sys((unsigned*)(a.ws[(rank + p) & 7] + fo), call);
                    }
                }
                if (!more) break;
#pragma unroll
                for (int u = 0; u < 4; ++u) r[u] = rn[u];
            }
        } else {
            const int cp = c - 8, dl = 8 - dd;
            const __amdgpu_buffer_rsrc_t rP = mk_rsrc(a.ws[peer], WS_NEED);
            unsigned qoff = QC_OFF + (unsigned)(dd * 8 + wid) * 64u;
            asm volatile("" : "+v"(qoff));
            unsigned* const qc = (unsigned*)(myws + qoff);
            const unsigned qbase = (call - 1u) * (unsigned)NPULL - (unsigned)NPW;
            const unsigned* const sf = (const unsigned*)(myws + SF_OFF) + peer * 192 + lane;
            int su = NPUSHED;
            unsigned f0, f1, f2;
            auto sf_issue = [&]() { f0 = ld_sys(sf); f1 = ld_sys(sf + 64); f2 = ld_sys(sf + 128); };
            auto sf_eval = [&]() {
                const unsigned long long m0 = ~__builtin_amdgcn_ballot_w64((int)(f0 - call) >= 0 || lane < NPUSHED);
                const unsigned long long m1 = ~__builtin_amdgcn_ballot_w64((int)(f1 - call) >= 0);
                const unsigned long long m2 = ~__builtin_amdgcn_ballot_w64((int)(f2 - call) >= 0 || lane >= NUNITS - 128);
                su = m0 ? __builtin_ctzll(m0) : m1 ? 64 + __builtin_ctzll(m1) : m2 ? 128 + __builtin_ctzll(m2) : 192;
            };
            auto sf_wait = [&](int n) {
                if (n < su) return;
                const unsigned long long t0 = __builtin_amdgcn_s_memrealtime();
                for (;;) {
                    sf_issue(); sf_eval();
                    if (n < su) break;
                    if ((unsigned)__builtin_amdgcn_readfirstlane((int)ld_agent(tmo)) == call ||
                        __builtin_amdgcn_s_memrealtime() - t0 > DEADLINE_TICKS) {
                        if (lane == 0) st_agent(tmo, call);
                        su = 192; break;
                    }
                    __builtin_amdgcn_s_sleep(2);
                }
            };
            auto unit = [&](int q, int& n, unsigned& so, unsigned& dsto, unsigned& flo) {
                n = NPUSHED + q;
                int kind, step, idx; decode_unit(cu, n, dl, kind, step, idx);
                so = SR_OFF + par * SR_BYTES + src_slot(kind, step, idx, dl) * UNIT;
                if (kind == 0) {
                    dsto = WD_OFF + par * WD_BYTES + (unsigned)((peer * 8 + step) * 8 + idx) * UNIT;
                    flo = WF_OFF + (unsigned)(idx * 64 + peer * 8 + step) * 16u + (unsigned)wid * 2u;
                } else {
                    const int tm = ((peer + 1 + idx) & 7) * 2 + (kind - 1);
                    dsto = XD_OFF + (unsigned)((peer * 8 + step) * 16 + tm) * UNIT;
                    flo = XF_OFF + (unsigned)(tm * 64 + peer * 8 + step) * 16u + (unsigned)wid * 2u;
                }
            };
            auto set_flag = [&](const __amdgpu_buffer_rsrc_t rs, unsigned flo) {
                if (lane == 0) __builtin_amdgcn_raw_buffer_store_b16((unsigned short)call, rs, 0, flo, 17);
            };
            const unsigned poff = WD_OFF + par * WD_BYTES + (unsigned)(rank * 64 + pu) * UNIT;
            store16<17>(r[0], rP, voffS, poff); WAIT_V(0);
            store16<17>(r[1], rP, voffS, poff + 8192u); WAIT_V(0);
            store16<17>(r[2], rP, voffS, poff + 16384u); sf_issue(); WAIT_V(0);
            store16<17>(r[3], rP, voffS, poff + 24576u);
            sf_eval();
            const unsigned flpush = WF_OFF + (unsigned)((pu & 7) * 64 + rank * 8 + (pu >> 3)) * 16u + (unsigned)wid * 2u;
            if (cp >= NPW) {
                WAIT_V(0);
                set_flag(rP, flpush);
            } else {
            int n0, n1 = 0; unsigned so0, do0, fl0, so1 = 0, do1 = 0, fl1 = 0, flp = 0;
            unit(cp, n0, so0, do0, fl0);
            sf_wait(n0);
            u32x4 v = __builtin_amdgcn_raw_buffer_load_b128(rP, voffS, so0, 17);
            bool pend = false;
            unsigned qv = 0;
#pragma unroll 1
            for (int k = 0;; ++k) {
                bool more = false;
#pragma unroll
                for (int u = 0; u < 4; ++u) {
                    WAIT_V(0);
                    asm volatile("" : "+v"(v));
                    const u32x4 w = v;
                    if (u < 3) v = __builtin_amdgcn_raw_buffer_load_b128(rP, voffS, so0 + (unsigned)(u + 1) * 8192u, 17);
                    else if (more) {
                        if (pend) { sf_eval(); pend = false; }
                        sf_wait(n1);
                        v = __builtin_amdgcn_raw_buffer_load_b128(rP, voffS, so1, 17);
                        if (su < NUNITS) { sf_issue(); pend = true; }
                    }
                    store16<17>(w, rM, voffS, do0 + (unsigned)u * 8192u);
                    if (u == 0) {
                        if (k == 0) set_flag(rP, flpush);
                        else set_flag(rM, flp);
                        if (lane == 0) qv = __hip_atomic_fetch_add(qc, 1u, __ATOMIC_RELAXED, __HIP_MEMORY_SCOPE_AGENT);
                    }
                    if (u == 2) {
                        const int qn = __builtin_amdgcn_readfirstlane((int)(qv - qbase));
                        more = qn < NPULL;
                        if (more) unit(qn, n1, so1, do1, fl1);
                    }
                }
                flp = fl0;
                if (!more) break;
                so0 = so1; do0 = do1; fl0 = fl1;
            }
            WAIT_V(0);
            set_flag(rM, flp);
            }
        }
    } else {
        const int g = b - NCOPY, xcd = g & 7, idx = g >> 3;
        mi = 4 * (xcd >> 1) + (idx & 3); nj = 4 * (xcd & 1) + (idx >> 2); colrank = rank;
    }
    const int rb = mi >> 1, hh = mi & 1;

    const int wr = wid >> 2, wc = wid & 3, fr = lane & 15, fq = lane >> 4;
    const unsigned ldsw = (unsigned)wid * 1024u;
    const int aoff = lds_byte(wr * 64 + fr, fq * 8);
    unsigned boff[2][2];
    { const int q = fr >> 2, p = lane & 3;
#pragma unroll
      for (int n = 0; n < 2; ++n)
#pragma unroll
          for (int t = 0; t < 2; ++t) {
              const int ch = 4 * wc + 2 * n + (p >> 1), f = (q << 2) | ((2 * fq + t) & 3);
              boff[n][t] = (unsigned)(size_t)lds + 65536u + 256u * (unsigned)(8 * fq + 4 * t + q) + 16u * (unsigned)(ch ^ f) + 8u * (unsigned)(p & 1);
          } }
#define SA(bb, h) (((bb) * 2 + (h)) * HTB)
#define SB(bb, h) ((4 + (bb) * 2 + (h)) * HTB)
#define SBR(bb, h) (((bb) * 2 + (h)) * HTB)
#define STAGE(bufoff, rs, voff, soff, istr) do { _Pragma("unroll") for (int _i = 0; _i < 2; ++_i) \
        __builtin_amdgcn_raw_ptr_buffer_load_lds(rs, (LAS void*)(lds + (bufoff) + ldsw + _i * 8192), 16, voff, (soff) + _i * (istr), 0, 16); } while (0)
#define LDA(dst, bb, h) do { _Pragma("unroll") for (int m = 0; m < 4; ++m) _Pragma("unroll") for (int k = 0; k < 2; ++k) \
        dst[m][k] = *(const LAS bf16x8*)(lds + SA(bb, h) + aoff + m * 2048 + k * 1024); } while (0)
#define TRR(d, addr, off) asm volatile("ds_read_b64_tr_b16 %0, %1 offset:%2" : "=&v"(d) : "v"(addr), "i"(off) : "memory")
#define LDB(dst, bb, h) do { _Pragma("unroll") for (int n = 0; n < 2; ++n) _Pragma("unroll") for (int k = 0; k < 2; ++k) { \
        TRR(dst[n][k][0], boff[n][0], SBR(bb, h) + k * 8192); TRR(dst[n][k][1], boff[n][1], SBR(bb, h) + k * 8192); } } while (0)
#define PKB(v) __builtin_shufflevector((v)[0], (v)[1], 0, 1, 2, 3, 4, 5, 6, 7)
#define MMA(ai, bj, At, Bt) do { SCHED; __builtin_amdgcn_s_setprio(1); _Pragma("unroll") for (int m = 0; m < 4; ++m) _Pragma("unroll") for (int n = 0; n < 2; ++n) \
        _Pragma("unroll") for (int k = 0; k < 2; ++k) acc[ai][bj][m][n] = __builtin_amdgcn_mfma_f32_16x16x32_bf16(PKB(Bt[n][k]), At[m][k], acc[ai][bj][m][n], 0, 0, 0); \
        __builtin_amdgcn_s_setprio(0); } while (0)

    f32x4 acc[2][2][4][2];
#pragma unroll
    for (int i0 = 0; i0 < 2; ++i0)
#pragma unroll
        for (int i1 = 0; i1 < 2; ++i1)
#pragma unroll
            for (int m = 0; m < 4; ++m)
#pragma unroll
                for (int n = 0; n < 2; ++n) acc[i0][i1][m][n] = (f32x4){0.f, 0.f, 0.f, 0.f};
    bf16x8 At[4][2]; s16x4 B0[2][2][2], B1[2][2][2];

    const int np = (rb == rank) ? 7 : 6;
    const unsigned long long valid = ~((0xFFull << (rank * 8)) | (0xFFull << (rb * 8)));
    const int nph = !is_copy || c < 8 ? 2 : 0;
    const unsigned dgo = DG_OFF + (unsigned)(b % 56) * 262144u + (unsigned)wid * 32768u;
    unsigned* const dgf = (unsigned*)(myws + DF_OFF) + (b % 56) * 8 + wid;
#pragma unroll 1
    for (int ph = 0; ph < nph; ++ph) {
        const bool own = is_copy || ph == 0;
        const int mic = is_copy ? rank * 2 + 1 - ph : mi;
        const int nt = own ? 8 : np * 8;
        const __amdgpu_buffer_rsrc_t rA = mk_rsrc(own ? a.x : (const char*)myws, own ? X_BYTES : WS_NEED);
        const __amdgpu_buffer_rsrc_t rB = mk_rsrc(own ? a.w : (const char*)myws, own ? W_BYTES : WS_NEED);
        const unsigned vA = own ? voffA : voffS, vB = own ? voffB : voffS;
        const unsigned iA = own ? 64u * XP : 8192u, iB = own ? 32u * WP : 8192u;
        const unsigned hA = own ? 128u * XP : HTB, hB = own ? 256u : HTB;
        auto addrA = [&](int id) -> unsigned {
            return own ? (unsigned)mic * 256u * XP + (unsigned)id * 128u
                       : XD_OFF + (unsigned)(id * 16 + mi) * UNIT;
        };
        auto addrB = [&](int id) -> unsigned {
            return own ? (unsigned)id * 64u * WP + (unsigned)(colrank * 2048 + nj * 256) * 2u
                       : WD_OFF + par * WD_BYTES + (unsigned)(id * 8 + nj) * UNIT;
        };
        unsigned long long ready = own ? 0xFFull : 0ull, seen = ready;
        const unsigned xf = XF_OFF + (unsigned)(mi * 64) * 16u, wf = WF_OFF + (unsigned)(nj * 64) * 16u;
        LAS volatile unsigned* const lavail = (LAS volatile unsigned*)(lds + 131072);
        int pslot = 0;
        const short c16 = (short)call;
        const s16x8 c8 = {c16, c16, c16, c16, c16, c16, c16, c16};
        auto arrived = [&](const u32x4 fx, const u32x4 fw) -> unsigned long long {
            const s16x8 dx = __builtin_bit_cast(s16x8, fx) - c8, dw = __builtin_bit_cast(s16x8, fw) - c8;
            const u32x4 dd4 = __builtin_bit_cast(u32x4, dx) | __builtin_bit_cast(u32x4, dw);
            const bool ok = ((dd4.x | dd4.y | dd4.z | dd4.w) & 0x80008000u) == 0u;
            return __builtin_amdgcn_ballot_w64(ok) & valid & ~seen;
        };
        auto pop = [&]() -> int { const int id = __builtin_ctzll(ready); ready &= ready - 1ull; return id; };
        int remaining = nt;
#pragma unroll 1
        while (remaining > 0) {
#pragma unroll 1
            while (__builtin_popcountll(ready) < 2) {
                if (wid == 0) {
                    const unsigned long long t0 = __builtin_amdgcn_s_memrealtime();
                    unsigned long long mask;
                    for (;;) {
                        CFENCE;
                        const u32x4 fx = __builtin_amdgcn_raw_buffer_load_b128(rM, (unsigned)lane * 16u, xf, 17);
                        const u32x4 fw = __builtin_amdgcn_raw_buffer_load_b128(rM, (unsigned)lane * 16u, wf, 17);
                        mask = arrived(fx, fw);
                        if (mask != 0ull) break;
                        if ((unsigned)__builtin_amdgcn_readfirstlane((int)ld_agent(tmo)) == call ||
                            __builtin_amdgcn_s_memrealtime() - t0 > DEADLINE_TICKS) {
                            if (lane == 0) st_agent(tmo, call);
                            mask = valid & ~seen; break;
                        }
                        __builtin_amdgcn_s_sleep(2);
                    }
                    if (lane == 0) { lavail[pslot] = (unsigned)mask; lavail[pslot + 1] = (unsigned)(mask >> 32); }
                    WAIT_V(0); WAIT_L(0);
                }
                CFENCE; BAR; CFENCE;
                const unsigned lo = (unsigned)__builtin_amdgcn_readfirstlane((int)lavail[pslot]), hi = (unsigned)__builtin_amdgcn_readfirstlane((int)lavail[pslot + 1]);
                pslot ^= 2;
                const unsigned long long m = ((unsigned long long)hi << 32) | lo;
                ready |= m; seen |= m;
            }
            const int id0 = pop(), id1 = pop();
            { const unsigned a0 = addrA(id0), b0 = addrB(id0), a1 = addrA(id1), b1 = addrB(id1);
              STAGE(SB(0, 0), rB, vB, b0, iB); STAGE(SA(0, 0), rA, vA, a0, iA); STAGE(SB(0, 1), rB, vB, b0 + hB, iB); STAGE(SA(0, 1), rA, vA, a0 + hA, iA);
              if (wr == 1) BAR;
              WAIT_V(4); BAR;
              STAGE(SB(1, 0), rB, vB, b1, iB); STAGE(SA(1, 0), rA, vA, a1, iA); STAGE(SB(1, 1), rB, vB, b1 + hB, iB);
              WAIT_V(6); BAR; }
            unsigned a1 = addrA(id1);
#pragma unroll 1
            for (;;) {
                remaining -= 2;
                const bool last = __builtin_popcountll(ready) < 2;
                const int t2 = last ? id0 : pop();
                const int t3 = last ? id0 : pop();
                const bool look = !own && (valid & ~seen) != 0ull;
                const unsigned a2 = addrA(t2), b2 = addrB(t2);
                const unsigned a3 = addrA(t3), b3 = addrB(t3);
                u32x4 fx, fw;
                LDB(B0, 0, 0); SCHED; LDA(At, 0, 0); STAGE(SA(1, 1), rA, vA, a1 + hA, iA);
                WAIT_L(8); BAR; WAIT_L(0); MMA(0, 0, At, B0); BAR; SCHED;
                LDB(B1, 0, 1); STAGE(SB(0, 0), rB, vB, b2, iB);
                BAR; WAIT_L(0); MMA(0, 1, At, B1); BAR;
                LDA(At, 0, 1); STAGE(SA(0, 0), rA, vA, a2, iA);
                BAR; WAIT_L(0); MMA(1, 0, At, B0); BAR; SCHED;
                STAGE(SB(0, 1), rB, vB, b2 + hB, iB);
                WAIT_V(6); BAR; MMA(1, 1, At, B1); BAR;
                if (look && wid == 0) {
                    fx = __builtin_amdgcn_raw_buffer_load_b128(rM, (unsigned)lane * 16u, xf, 17);
                    fw = __builtin_amdgcn_raw_buffer_load_b128(rM, (unsigned)lane * 16u, wf, 17);
                }
                LDB(B0, 1, 0); SCHED; LDA(At, 1, 0); STAGE(SA(0, 1), rA, vA, a2 + hA, iA);
                WAIT_L(8); BAR; WAIT_L(0); MMA(0, 0, At, B0); BAR; SCHED;
                LDB(B1, 1, 1); STAGE(SB(1, 0), rB, vB, b3, iB);
                BAR; WAIT_L(0); MMA(0, 1, At, B1); BAR;
                LDA(At, 1, 1); STAGE(SA(1, 0), rA, vA, a3, iA);
                BAR; WAIT_L(0); MMA(1, 0, At, B0); BAR; SCHED;
                STAGE(SB(1, 1), rB, vB, b3 + hB, iB);
                WAIT_V(6);
                if (look && wid == 0) {
                    const unsigned long long mask = arrived(fx, fw);
                    if (lane == 0) { lavail[4] = (unsigned)mask; lavail[5] = (unsigned)(mask >> 32); }
                    WAIT_L(0);
                }
                CFENCE; BAR; CFENCE;
                unsigned lo = 0, hi = 0;
                if (look) { lo = lavail[4]; hi = lavail[5]; }
                MMA(1, 1, At, B1); BAR;
                if (look) {
                    const unsigned long long m = ((unsigned long long)(unsigned)__builtin_amdgcn_readfirstlane((int)hi) << 32) | (unsigned)__builtin_amdgcn_readfirstlane((int)lo);
                    ready |= m; seen |= m;
                }
                a1 = a3;
                if (last) break;
            }
            WAIT_V(0);
            if (wr == 0) BAR;
            BAR;
        }
        if (is_copy && ph == 0) {
#pragma unroll
            for (int u = 0; u < 32; ++u) {
                f32x4& v = acc[u >> 4][(u >> 3) & 1][(u >> 1) & 3][u & 1];
                store16<17>(__builtin_bit_cast(u32x4, v), rM, (unsigned)lane * 16u, dgo + (unsigned)u * 1024u);
                v = (f32x4){0.f, 0.f, 0.f, 0.f};
            }
            WAIT_V(0);
            if (lane == 0) st_agent(dgf, call);
        }
    }
    if (nph == 0) {
        const unsigned long long t0 = __builtin_amdgcn_s_memrealtime();
        while ((unsigned)__builtin_amdgcn_readfirstlane((int)ld_agent(dgf)) != call) {
            if ((unsigned)__builtin_amdgcn_readfirstlane((int)ld_agent(tmo)) == call ||
                __builtin_amdgcn_s_memrealtime() - t0 > DEADLINE_TICKS) {
                if (lane == 0) st_agent(tmo, call);
                break;
            }
            __builtin_amdgcn_s_sleep(2);
        }
        CFENCE;
#pragma unroll
        for (int u = 0; u < 32; ++u)
            acc[u >> 4][(u >> 3) & 1][(u >> 1) & 3][u & 1] =
                __builtin_bit_cast(f32x4, __builtin_amdgcn_raw_buffer_load_b128(rM, (unsigned)lane * 16u, dgo + (unsigned)u * 1024u, 17));
    }

    const bool bad0 = (unsigned)__builtin_amdgcn_readfirstlane((int)ld_agent(tmo)) == call;
    const u32x4 nan4 = {0x7FC07FC0u, 0x7FC07FC0u, 0x7FC07FC0u, 0x7FC07FC0u};
    constexpr unsigned PSTR = 8192u;
    const unsigned pbase = PS_OFF + par * PS_BYTES + (unsigned)(mi * 8 + nj) * 131072u + (unsigned)wid * 1024u;
    if (!is_copy && rb != rank) {
#pragma unroll
        for (int ai = 0; ai < 2; ++ai)
#pragma unroll
            for (int bj = 0; bj < 2; ++bj)
#pragma unroll
                for (int m = 0; m < 4; ++m) {
                    const f32x4 v0 = acc[ai][bj][m][0], v1 = acc[ai][bj][m][1];
                    u32x4 w; w.x = pack_bf16(v0[0], v0[1]); w.y = pack_bf16(v0[2], v0[3]); w.z = pack_bf16(v1[0], v1[1]); w.w = pack_bf16(v1[2], v1[3]);
                    if (bad0) w = nan4;
                    store16<17>(w, rM, (unsigned)lane * 16u, pbase + (unsigned)((ai * 2 + bj) * 4 + m) * PSTR);
                }
        WAIT_V(0);
        if (lane == 0) st_sys((unsigned*)(a.ws[rb] + PF_OFF) + (unsigned)((rank * 2 + hh) * 8 + nj) * 32u + wid, call);
        return;
    }
    const __amdgpu_buffer_rsrc_t rO = mk_rsrc(a.out, O_BYTES);
    const unsigned vo = (unsigned)(wr * 64 + fr) * WP + (unsigned)(wc * 32 + ((fq & 1) << 4) + ((fq >> 1) << 3)) * 2u;
    const unsigned so = (unsigned)hh * 256u * WP + (unsigned)(colrank * 2048 + nj * 256) * 2u;
    auto store_out = [&](int ai, int bj, int m, bool bad) {
        const f32x4 v0 = acc[ai][bj][m][0], v1 = acc[ai][bj][m][1];
        const unsigned ax = pack_bf16(v0[0], v0[1]), ay = pack_bf16(v0[2], v0[3]);
        const unsigned bx = pack_bf16(v1[0], v1[1]), by = pack_bf16(v1[2], v1[3]);
        const auto sx = __builtin_amdgcn_permlane16_swap(ax, bx, false, false);
        const auto sy = __builtin_amdgcn_permlane16_swap(ay, by, false, false);
        u32x4 w = {sx[0], sy[0], sx[1], sy[1]};
        if (bad) w = nan4;
        store16<OUT_AUX>(w, rO, vo, so + (unsigned)(ai * 128 + m * 16) * WP + (unsigned)bj * 256u);
    };
    if (is_copy) {
        const unsigned* const pf = (const unsigned*)(myws + PF_OFF) + (unsigned)((colrank * 2 + hh) * 8 + nj) * 32u + wid;
        const unsigned long long t0 = __builtin_amdgcn_s_memrealtime();
        bool bad = bad0;
        while ((int)((unsigned)__builtin_amdgcn_readfirstlane((int)ld_sys(pf)) - call) < 0) {
            if ((unsigned)__builtin_amdgcn_readfirstlane((int)ld_agent(tmo)) == call ||
                __builtin_amdgcn_s_memrealtime() - t0 > DEADLINE_TICKS) {
                if (lane == 0) st_agent(tmo, call);
                bad = true;
                break;
            }
            __builtin_amdgcn_s_sleep(2);
        }
        CFENCE;
        const __amdgpu_buffer_rsrc_t rP = mk_rsrc(a.ws[colrank], WS_NEED);
        auto combine = [&](int u, const u32x4 w) {
            const int ai = u >> 3, bj = (u >> 2) & 1, m = u & 3;
            f32x4& v0 = acc[ai][bj][m][0]; f32x4& v1 = acc[ai][bj][m][1];
            v0[0] += __uint_as_float(w.x << 16); v0[1] += __uint_as_float(w.x & 0xFFFF0000u);
            v0[2] += __uint_as_float(w.y << 16); v0[3] += __uint_as_float(w.y & 0xFFFF0000u);
            v1[0] += __uint_as_float(w.z << 16); v1[1] += __uint_as_float(w.z & 0xFFFF0000u);
            v1[2] += __uint_as_float(w.w << 16); v1[3] += __uint_as_float(w.w & 0xFFFF0000u);
            store_out(ai, bj, m, bad);
        };
        auto pull = [&](auto winc) {
            constexpr int W = decltype(winc)::value;
            u32x4 pv[W];
#pragma unroll
            for (int j = 0; j < W; ++j) pv[j] = __builtin_amdgcn_raw_buffer_load_b128(rP, (unsigned)lane * 16u, pbase + (unsigned)j * PSTR, 17);
#pragma unroll
            for (int u = 0; u < 16; ++u) {
                const int nl = (u + W - 1 < 16 ? W - 1 : 15 - u), ns = u < W ? u : W;
                switch (nl + ns) {
                    case 0: WAIT_V(0); break; case 1: WAIT_V(1); break; case 2: WAIT_V(2); break; case 3: WAIT_V(3); break;
                    case 4: WAIT_V(4); break; case 5: WAIT_V(5); break; case 6: WAIT_V(6); break; default: WAIT_V(7); break;
                }
                asm volatile("" : "+v"(pv[u % W]));
                const u32x4 w = pv[u % W];
                if (u + W < 16) pv[u % W] = __builtin_amdgcn_raw_buffer_load_b128(rP, (unsigned)lane * 16u, pbase + (unsigned)(u + W) * PSTR, 17);
                SCHED;
                combine(u, w);
                SCHED;
            }
        };
        if (hh == 0) pull(std::integral_constant<int, WINU>{}); else pull(std::integral_constant<int, WINL>{});
    } else {
#pragma unroll
        for (int ai = 0; ai < 2; ++ai)
#pragma unroll
            for (int m = 0; m < 4; ++m)
#pragma unroll
                for (int bj = 0; bj < 2; ++bj) store_out(ai, bj, m, bad0);
    }
}

}

extern "C" void kernel_launch(int rank, int nranks, ncclComm_t comm,
                              hipStream_t stream,
                              void* const* d_in, const int* in_sizes, int n_in,
                              void* d_out, int out_size,
                              const int* in_dtypes, int out_dtype,
                              void* const* d_out_all,
                              void* const* d_ws_all, size_t ws_bytes) {
    (void)comm; (void)in_dtypes; (void)out_dtype; (void)d_out_all;
    if (nranks != NR || n_in < 2 || rank < 0 || rank >= NR || ws_bytes < (size_t)WS_NEED ||
        in_sizes[0] != 4096 * 512 || in_sizes[1] != 512 * 16384 || out_size != 512 * 16384) {
        fprintf(stderr, "kernel_launch: unexpected problem shape\n");
        return;
    }
    static bool attr_set[64] = {};
    int dev = 0;
    HIP_CHECK(hipGetDevice(&dev));
    if (dev >= 0 && dev < 64 && !attr_set[dev]) {
        HIP_CHECK(hipFuncSetAttribute((const void*)fused_gemm_rs, hipFuncAttributeMaxDynamicSharedMemorySize, (int)LDS_BYTES));
        attr_set[dev] = true;
    }
    Args a;
    a.x = (const char*)d_in[0];
    a.w = (const char*)d_in[1];
    a.out = (char*)d_out;
    for (int p = 0; p < NR; ++p) a.ws[p] = (char*)d_ws_all[p];
    a.rank = rank;
    hipLaunchKernelGGL(fused_gemm_rs, dim3(NBLK), dim3(512), LDS_BYTES, stream, a);
    HIP_CHECK(hipGetLastError());
}
```
